```python
import numpy as np

kernel_name = "gpu_ag_gemm_m4096_k4096_n16384_f32_reluq8"


ABSMAX_RTOL = 0.02

COMPUTE_DTYPE = "float32"
EPILOGUE = "relu_quant_int8"
IN_QUANT = None
OUT_QUANT = "int8"
IN_DTYPES = {"x": "float32", "w_mat": "float32"}
WORLD_SIZE = 8

def setup_inputs():

    x = np.random.randn(4096, 4096).astype(np.float32)
    w_mat = (np.random.randn(4096, 16384) / np.sqrt(4096)).astype(np.float32)
    return {"x": x, "w_mat": w_mat}

def _quant_dequant(y, amax):
    scale = amax / 127.0
    q = np.clip(np.round(y / scale), -127.0, 127.0)
    return (q * scale).astype(np.float32)
def _epilogue(y):
    y = np.maximum(y, 0.0)
    return _quant_dequant(y, np.abs(y).max())

def reference(x, w_mat):
    y = (x @ w_mat).astype(np.float32)
    return _epilogue(y)
```

```cpp
#include <hip/hip_runtime.h>
#include <rccl/rccl.h>
#include <cstdio>
#include <cstdint>

#define HIP_CHECK(expr) do { hipError_t _e = (expr); if (_e != hipSuccess) { \
    fprintf(stderr, "HIP error %s at %s:%d\n", hipGetErrorString(_e), __FILE__, __LINE__); \
    return; } } while (0)

namespace agk {
typedef unsigned u32x4 __attribute__((ext_vector_type(4)));
typedef unsigned u32x2 __attribute__((ext_vector_type(2)));
typedef float f32x4 __attribute__((ext_vector_type(4)));
typedef float f32x16 __attribute__((ext_vector_type(16)));
typedef unsigned long long u64;
#define LAS __attribute__((address_space(3)))
#define GAS __attribute__((address_space(1)))

constexpr int NR = 8, KT = 4096, NLOC = 2048;
constexpr int NST = 128;
constexpr int RING = 12, ST_A = 16384, ST_B = 10240;
constexpr int XD = 3;
constexpr int NT = 13;
constexpr int NWG = 256, NGEMM = 208, NUNIT = 512;
constexpr int NPULL = 32;
constexpr int KP = 12;
constexpr int NPUSH = 12;
constexpr int NDIRECT = 128;
constexpr int SPEC = 50;
constexpr int SPECR = 7;
constexpr int ROUNDS = 2;
constexpr int PP0 = 2400;
constexpr int PP1 = 3500;
constexpr int TEND = 5100;
constexpr int PQ1 = 76;
constexpr int PQS = 32;
constexpr int PQF = 1;
constexpr int PZ = 3800;
constexpr int XDL = 60;
constexpr int WDL1 = 80;
constexpr int WDL = 250;
constexpr int NPREP = NWG - NGEMM - NPULL;
constexpr int W1 = 4;
constexpr int XSKEW = 36864;
constexpr unsigned XTS = NST * ST_A + XSKEW;
constexpr int PKS = 5888;
static_assert(NPREP >= 2 && NPUSH <= NPREP && 7 * KP <= 6 * NPULL && NPULL % 7 != 0 && NPULL >= 8, "roles");

constexpr u64 DEADLINE = 50000000ull;
constexpr unsigned BIG = 0x40000000u;

constexpr size_t WS_CNT = 0;
constexpr size_t WS_TMO = 8192;
constexpr size_t WS_PZ  = 10240;
constexpr size_t WS_TK  = 12288;
constexpr size_t WS_AML = 16384;
constexpr size_t WS_AMG = 24576;
constexpr size_t WS_FW  = 32768;
constexpr size_t WS_HDR = 65536;
constexpr size_t WS_FX  = WS_HDR + NR * NUNIT * 8;
constexpr size_t WS_DUM = 131072;
constexpr size_t WS_XB  = 1u << 20;
constexpr size_t WS_WB  = WS_XB + (36u << 20);
constexpr size_t WS_PK  = WS_WB + (17u << 20);
constexpr size_t WS_OV  = WS_PK + (3u << 20);
constexpr size_t WS_END = WS_OV + (2u << 20);
static_assert(WS_FX + 16384 + 256 <= WS_DUM && WS_DUM + 384 * 128 <= WS_XB && NUNIT * PKS <= (3u << 20) && (size_t)NT * NST * ST_B <= (17u << 20), "workspace map");

constexpr int L_CTRL = RING * ST_B;
constexpr int L_GO = L_CTRL;
constexpr int L_RDYB = L_CTRL + 16;
constexpr int L_WMAX = L_CTRL + 32, L_FIN = L_CTRL + 40;
constexpr int L_DUMMY = L_CTRL + 256;
constexpr int L_FREE = L_CTRL + 1024;
constexpr int L_PACK = L_CTRL + 2048;
constexpr int L_TOTAL = L_PACK + 4608;
static_assert(RING == 12 && ST_B == 10240, "literals in the main loop");

struct Params { const float* x; const float* w; char* own; int rank; int pad; char* ws[NR]; signed char* out; };

#define RLX_AGENT __ATOMIC_RELAXED, __HIP_MEMORY_SCOPE_AGENT
#define RLX_SYS __ATOMIC_RELAXED, __HIP_MEMORY_SCOPE_SYSTEM
#define VM0() asm volatile("s_waitcnt vmcnt(0)" ::: "memory")

__device__ __forceinline__ u64 rt() { return __builtin_amdgcn_s_memrealtime(); }
__device__ __forceinline__ unsigned rfl(unsigned v) { return __builtin_amdgcn_readfirstlane(v); }
__device__ __forceinline__ void st16(void* p, u32x4 v) {
    asm volatile("global_store_dwordx4 %0, %1, off sc0 sc1\n\ts_nop 1" :: "v"(p), "v"(v) : "memory");
}
__device__ __forceinline__ unsigned cvt2(float lo, float hi) {
    unsigned r; asm("v_cvt_pk_bf16_f32 %0, %1, %2" : "=v"(r) : "v"(lo), "v"(hi)); return r;
}
__device__ __forceinline__ unsigned lds_ld(unsigned addr) {
    unsigned v; asm volatile("ds_read_b32 %0, %1\n\ts_waitcnt lgkmcnt(0)" : "=v"(v) : "v"(addr) : "memory"); return rfl(v);
}
__device__ __forceinline__ void lds_st(unsigned addr, unsigned v) {
    asm volatile("ds_write_b32 %0, %1" :: "v"(addr), "v"(v) : "memory");
}
__device__ __forceinline__ unsigned wave_max_u32(unsigned v) {
    v = max(v, (unsigned)__builtin_amdgcn_update_dpp(0, (int)v, 0x111, 0xf, 0xf, true));
    v = max(v, (unsigned)__builtin_amdgcn_update_dpp(0, (int)v, 0x112, 0xf, 0xf, true));
    v = max(v, (unsigned)__builtin_amdgcn_update_dpp(0, (int)v, 0x114, 0xf, 0xf, true));
    v = max(v, (unsigned)__builtin_amdgcn_update_dpp(0, (int)v, 0x118, 0xf, 0xf, true));
    v = max(v, (unsigned)__builtin_amdgcn_update_dpp(0, (int)v, 0x142, 0xa, 0xf, false));
    v = max(v, (unsigned)__builtin_amdgcn_update_dpp(0, (int)v, 0x143, 0xc, 0xf, false));
    return (unsigned)__builtin_amdgcn_readlane((int)v, 63);
}
__device__ __forceinline__ unsigned wave_scan_u32(unsigned v) {
    v += (unsigned)__builtin_amdgcn_update_dpp(0, (int)v, 0x111, 0xf, 0xf, true);
    v += (unsigned)__builtin_amdgcn_update_dpp(0, (int)v, 0x112, 0xf, 0xf, true);
    v += (unsigned)__builtin_amdgcn_update_dpp(0, (int)v, 0x114, 0xf, 0xf, true);
    v += (unsigned)__builtin_amdgcn_update_dpp(0, (int)v, 0x118, 0xf, 0xf, true);
    v += (unsigned)__builtin_amdgcn_update_dpp(0, (int)v, 0x142, 0xa, 0xf, false);
    v += (unsigned)__builtin_amdgcn_update_dpp(0, (int)v, 0x143, 0xc, 0xf, false);
    return v;
}
__device__ __forceinline__ bool give_up(unsigned* tmo, unsigned call, u64 t0, unsigned& spins) {
    if ((++spins & 63u) != 0u) return false;
    if (rfl(__hip_atomic_load(tmo, RLX_AGENT)) == call || rt() - t0 > DEADLINE) {
        __hip_atomic_store(tmo, call, RLX_AGENT);
        VM0();
        return true;
    }
    return false;
}
constexpr unsigned WAHEAD = 14;

__device__ __forceinline__ char* lane_peer(const Params& p, int lane) {
    u64 r = 0;
#pragma unroll
    for (int i = 0; i < NR; ++i) { u64 q = (u64)p.ws[i]; asm("" : "+s"(q)); r = (lane == i) ? q : r; }
    return (char*)(__attribute__((address_space(1))) char*)r;
}

__device__ __forceinline__ void conv_w(const float* w, char* ws, int nt, int ks, int lane, const unsigned* cnt) {
    typedef float f32x2 __attribute__((ext_vector_type(2)));
    const unsigned callm1 = __hip_atomic_load(cnt, RLX_AGENT);
    char* dst = ws + WS_WB + (size_t)(nt * NST + ks) * ST_B + lane * 16;
    const float* src = w + (size_t)(ks * 32 + (lane >> 4) * 8) * NLOC + nt * 128;
    f32x4 v[2][8]; f32x2 t[8];
#pragma unroll
    for (int i = 0; i < 16; ++i) {
        const int h = i >> 3, e = i & 7;
        v[h][e] = *(const f32x4*)(src + (size_t)e * NLOC + (lane & 15) * 4 + h * 64);
        if (i & 1) __builtin_amdgcn_sched_barrier(0);
    }
#pragma unroll
    for (int e = 0; e < 8; ++e) t[e] = *(const f32x2*)(src + (size_t)e * NLOC + (lane & 15) * 2 + (nt < NT - 1 ? 1664 - 96 * nt : 0));
#pragma unroll
    for (int h = 0; h < 2; ++h) {
        __builtin_amdgcn_sched_barrier(0);
#pragma unroll
        for (int c = 0; c < 4; ++c)
            st16(dst + (h * 4 + c) * 1024, (u32x4){cvt2(v[h][0][c], v[h][1][c]), cvt2(v[h][2][c], v[h][3][c]), cvt2(v[h][4][c], v[h][5][c]), cvt2(v[h][6][c], v[h][7][c])});
    }
    __builtin_amdgcn_sched_barrier(0);
#pragma unroll
    for (int c = 0; c < 2; ++c)
        st16(dst + (8 + c) * 1024, (u32x4){cvt2(t[0][c], t[1][c]), cvt2(t[2][c], t[3][c]), cvt2(t[4][c], t[5][c]), cvt2(t[6][c], t[7][c])} & (nt < NT - 1 ? ~0u : 0u));
    VM0();
    if (lane == 0) __hip_atomic_store((unsigned*)(ws + WS_FW) + nt * NST + ks, callm1 + 1u, RLX_SYS);
}

template <int NH>
__device__ __forceinline__ void pack_x(const Params& p, char* ws, int idx, int h0, int lane, const unsigned* cnt, LAS unsigned char* sc) {
    const unsigned callm1 = __hip_atomic_load(cnt, RLX_AGENT);
    typedef unsigned short u16x2 __attribute__((ext_vector_type(2)));
    typedef unsigned u32x3 __attribute__((ext_vector_type(3)));
    const int rank = p.rank, ks = idx >> 2, rp = idx & 3;
    const float* src = p.x + (size_t)(rp * 128 + h0 * 64 + (lane & 15)) * KT + ks * 32 + (lane >> 4) * 8;
    f32x4 v[8 * NH];
#pragma unroll
    for (int f = 0; f < 4 * NH; ++f) {
        v[2 * f] = *(const f32x4*)(src + (size_t)f * 16 * KT);
        v[2 * f + 1] = *(const f32x4*)(src + (size_t)f * 16 * KT + 4);
    }
    char* pw = lane_peer(p, lane);
    asm volatile("" : "+v"(pw));
    unsigned D[16 * NH];
#pragma unroll
    for (int i = 0; i < 8 * NH; ++i) { D[2 * i] = cvt2(v[i].x, v[i].y); D[2 * i + 1] = cvt2(v[i].z, v[i].w); }
    const unsigned slot = ((rank * 2 + (rp >> 1)) * NST + ks) * 2 + (rp & 1);
    const size_t xoff = WS_XB + (size_t)(rank * 2 + (rp >> 1)) * XTS + (ks * 2 + (rp & 1)) * 8192 + h0 * 4096 + lane * 16;
    const bool push = NH == 1 && idx < NPUSH;
    for (int i = push ? 1 : NR; i <= NR; ++i) {
        char* xr = p.ws[(rank + ((i + idx) % 7 + 1) * (i < NR)) & 7] + xoff;
#pragma unroll
        for (int f = 0; f < 4 * NH; ++f) st16(xr + f * 1024, (u32x4){D[4 * f], D[4 * f + 1], D[4 * f + 2], D[4 * f + 3]});
    }
    if (push) {
        asm volatile("s_barrier\n\ts_waitcnt vmcnt(0)\n\ts_barrier" ::: "memory");
        if (h0 == 0 && lane < NR) __hip_atomic_store((GAS unsigned*)(pw + WS_FX) + slot, callm1 + 1u, RLX_SYS);
        return;
    }
    float m0 = 0.f, m1 = 0.f;
#pragma unroll
    for (int i = 0; i < 8 * NH; ++i) asm("v_max3_f32 %0, %0, |%2|, |%3|\n\tv_max3_f32 %1, %1, |%4|, |%5|" : "+v"(m0), "+v"(m1) : "v"(v[i].x), "v"(v[i].y), "v"(v[i].z), "v"(v[i].w));
    const unsigned el = max(__float_as_uint(m0), __float_as_uint(m1)) >> 23 & 0xffu;
    unsigned top = 0;
#pragma unroll
    for (int bit = 128; bit; bit >>= 1) if (__builtin_popcountll(__ballot(el >= top + bit)) > 6 * NH) top += bit;
    const unsigned low = top > 7u ? top - 7u : 0u, sh = low & 1u, base = (low + sh) >> 1;
    const unsigned addc = (0x80u - base) * 0x01010101u;
    const u16x2 sh2 = {(unsigned short)(sh << 7), (unsigned short)(sh << 7)};
    char* pk = ws + WS_PK + (size_t)idx * PKS;
    unsigned H[8 * NH], M[2] = {0u, 0u};
#define BFI(m, a, b) (((m) & (a)) | (~(m) & (b)))
#pragma unroll
    for (int h = 0; h < NH; ++h) {
        unsigned L[8], d[8];
        u32x3 C = {0u, 0u, 0u};
#pragma unroll
        for (int i = 7; i >= 0; --i) {
            const unsigned a = __builtin_bit_cast(unsigned, __builtin_bit_cast(u16x2, D[h * 16 + 2 * i]) + sh2), c = __builtin_bit_cast(unsigned, __builtin_bit_cast(u16x2, D[h * 16 + 2 * i + 1]) + sh2);
            L[i] = __builtin_amdgcn_perm(c, a, 0x06040200u);
            H[h * 8 + i] = __builtin_amdgcn_perm(c, a, 0x07050301u);
            d[i] = (H[h * 8 + i] & 0x7f7f7f7fu) + addc;
            C[i >> 2] = BFI(0x03030303u, d[i], C[i >> 2] << 2);
            C[2] = BFI(0x80808080u, H[h * 8 + i], C[2] >> 1);
            M[h] = BFI(0x80808080u, (d[i] & 0x7c7c7c7cu) + 0x7f7f7f7fu, M[h] >> 1);
        }
        st16(pk + (h0 + h) * 2048 + lane * 16, (u32x4){L[0], L[1], L[2], L[3]});
        st16(pk + (h0 + h) * 2048 + 1024 + lane * 16, (u32x4){L[4], L[5], L[6], L[7]});
        asm volatile("global_store_dwordx3 %0, %1, off sc0 sc1\n\ts_nop 1" :: "v"(pk + 4096 + (h0 + h) * 768 + lane * 12), "v"(C) : "memory");
        *(volatile LAS u32x4*)(sc + (h0 + h) * 2048 + lane * 32) = (u32x4){d[0], d[1], d[2], d[3]};
        *(volatile LAS u32x4*)(sc + (h0 + h) * 2048 + lane * 32 + 16) = (u32x4){d[4], d[5], d[6], d[7]};
    }
#undef BFI
    const unsigned own = __builtin_popcount(M[0]) | (__builtin_popcount(M[1]) << 16), inc = wave_scan_u32(own);
    const unsigned tot = (unsigned)__builtin_amdgcn_readlane((int)inc, 63);
    unsigned c0 = tot & 0xffffu, c1 = tot >> 16, bs1 = base | (sh << 7);
    if (NH == 1) {
        volatile LAS unsigned* ex = (volatile LAS unsigned*)(sc + 4096 + 256);
        if (lane == 0) ex[h0] = c0 | bs1 << 16;
        asm volatile("s_waitcnt lgkmcnt(0)\n\ts_barrier" ::: "memory");
        const unsigned o = rfl(ex[h0 ^ 1]);
        c1 = h0 ? c0 : o & 0xffffu; c0 = h0 ? o & 0xffffu : c0; bs1 = o >> 16;
    }
    const unsigned n0 = (c0 + 1u) & ~1u, n = n0 + ((c1 + 1u) & ~1u);
    const bool raw = n > 128u;
    if (raw) {
        char* ov = ws + WS_OV + (size_t)idx * 4096 + h0 * 2048 + lane * 16;
#pragma unroll
        for (int q = 0; q < 2 * NH; ++q) st16(ov + q * 1024, (u32x4){H[4 * q], H[4 * q + 1], H[4 * q + 2], H[4 * q + 3]});
    } else {
        volatile LAS unsigned short* list = (volatile LAS unsigned short*)(sc + 4096);
        unsigned o0 = ((inc - own) & 0xffffu) + (h0 ? n0 : 0u), o1 = n0 + ((inc - own) >> 16);
        while (M[0] | M[1]) {
            if (M[0]) { const unsigned bt = __builtin_ctz(M[0]); M[0] &= M[0] - 1; list[o0++] = (unsigned short)(lane * 32 + (7 - (bt & 7)) * 4 + (bt >> 3)); }
            if (M[1]) { const unsigned bt = __builtin_ctz(M[1]); M[1] &= M[1] - 1; list[o1++] = (unsigned short)(lane * 32 + (7 - (bt & 7)) * 4 + (bt >> 3)); }
        }
        const unsigned e = lane * 2 + 1;
        if (e < n && (NH == 2 || (e > n0) == (h0 != 0))) {
            const unsigned w2 = *(volatile LAS unsigned*)(sc + 4096 + lane * 4);
            const unsigned p0 = w2 & 0x7ffu, p1 = (e == c0 || e == n0 + c1) ? p0 : (w2 >> 16 & 0x7ffu);
            volatile LAS unsigned char* pl = (volatile LAS unsigned char*)sc + (e > n0 ? 2048 : 0);
            const unsigned v0 = pl[p0], v1 = pl[p1];
            __hip_atomic_store((unsigned*)(pk + 5632) + lane, p0 | (v0 & 0x7cu) << 9 | p1 << 16 | (v1 & 0x7cu) << 25, RLX_SYS);
        }
    }
    const unsigned call = callm1 + 1u;
    u64 hd = ((u64)call << 32) | base | (sh << 7) | (raw ? 1u << 23 : (n0 << 7) | (n << 15)) | bs1 << 24;
    GAS u64* hq = (GAS u64*)(pw + WS_HDR) + rank * NUNIT + idx;
    asm volatile("s_waitcnt vmcnt(0)" : "+v"(hd), "+v"(hq) :: "memory");
    if (NH == 1) { asm volatile("s_barrier" ::: "memory"); if (h0) return; }
    if (lane < NR && (lane != rank) == (idx < NDIRECT)) __hip_atomic_store(hq, hd, RLX_SYS);
    if (lane == 0) __hip_atomic_store((unsigned*)(ws + WS_FX) + slot, call, RLX_SYS);
}

__device__ __forceinline__ void loader(const char* src, unsigned lbase, int lane) {
    const unsigned v0 = lane * 16, v1 = v0 + 4096, v2 = v0 + 8192;
    const unsigned vchk = lbase + (lane < 4 ? L_FREE + lane * 4 : L_RDYB);
    const unsigned vful = lbase + (lane == 0 ? L_GO + 4 : L_DUMMY + lane * 4);
    const unsigned slotend = lbase + RING * ST_B;
    unsigned keep, ss, pub, slot, t, vt;
#define G2(V) "global_load_lds_dwordx4 " V ", s[40:41] sc0 sc1\n\tglobal_load_lds_dwordx4 " V ", s[40:41] offset:1024 sc0 sc1\n\t"
#define G4(V) G2(V) "global_load_lds_dwordx4 " V ", s[40:41] offset:2048 sc0 sc1\n\tglobal_load_lds_dwordx4 " V ", s[40:41] offset:3072 sc0 sc1\n\t"
#define MNEXT "s_add_u32 m0, m0, 0x1000\n\ts_nop 0\n\t"
#define CHECK "ds_read_b32 %[vt], %[vchk]\n\ts_waitcnt lgkmcnt(0)\n\tv_cmp_lt_u32 vcc, %[ss], %[vt]\n\ts_cmp_eq_u64 vcc, -1\n\t"
#define PUB(W, J) "s_waitcnt vmcnt(" W ")\n\ts_add_i32 %[t], %[ss], " J "\n\ts_max_i32 %[pub], %[pub], %[t]\n\tv_mov_b32 %[vt], %[pub]\n\tds_write_b32 %[vful], %[vt]\n\t"
#define RETRY CHECK "s_cbranch_scc1 LI%=\n\t"
    asm volatile("s_mov_b32 %[k], m0\n\ts_mov_b64 s[40:41], %[g]\n\ts_mov_b32 %[ss], 0\n\ts_mov_b32 %[pub], 0\n\ts_mov_b32 %[slot], %[slot0]\n"
                 "LL%=:\n\t" CHECK "s_cbranch_scc0 LB%=\n"
                 "LI%=:\n\ts_mov_b32 m0, %[slot]\n\ts_add_u32 %[slot], %[slot], 10240\n\t"
                 G4("%[v0]") MNEXT G4("%[v1]") MNEXT G2("%[v2]")
                 "s_add_u32 s40, s40, 10240\n\ts_addc_u32 s41, s41, 0\n\ts_cmp_eq_u32 %[slot], %[slotend]\n\ts_cselect_b32 %[slot], %[slot0], %[slot]\n\ts_add_u32 %[ss], %[ss], 1\n\t"
                 PUB("20", "9")
                 "s_cmp_lt_u32 %[ss], 128\n\ts_cbranch_scc1 LL%=\n\t"
                 PUB("10", "10") PUB("0", "11") "s_branch LE%=\n"
                 "LB%=:\n\t" PUB("10", "10") RETRY PUB("0", "11")
                 "\nLS%=:\n\t" RETRY "s_sleep 4\n\ts_branch LS%=\nLE%=:\n\ts_waitcnt lgkmcnt(0)\n\ts_mov_b32 m0, %[k]"
                 : [k] "=&s"(keep), [ss] "=&s"(ss), [pub] "=&s"(pub), [slot] "=&s"(slot), [t] "=&s"(t), [vt] "=&v"(vt)
                 : [g] "s"(src), [slot0] "s"(lbase), [slotend] "s"(slotend), [vchk] "v"(vchk), [vful] "v"(vful), [v0] "v"(v0), [v1] "v"(v1), [v2] "v"(v2)
                 : "memory", "scc", "vcc", "s40", "s41");
#undef G2
#undef G4
#undef MNEXT
#undef CHECK
#undef PUB
#undef RETRY
}

#define POLL_CLOB "s64", "s65", "s66", "s67", "s68", "s69", "s70", "s71", "s72", "s73", "s74", "s75", "s76", "s77", "s78", "s79", "s80", "s81", "s82", "s83", "s84", "s85", "s86", "s87", "s88", "s89", "s90", "s91", "s92", "s93", "s94", "s95"
#define MAINLOOP \
        "s_mov_b32 %[ss], 31\n\t" \
        "s_mov_b32 %[sb], 0\n\t" \
        "s_mov_b64 s[40:41], %[xs]\n\t" \
        "v_mov_b32 %[vaw], %[vlw]\n\t" \
        "v_mov_b32 %[vt], 12\n\t" \
        "v_mov_b32 v0, 0\n\t" \
        "v_mov_b32 v1, 0\n\t" \
        "v_mov_b32 v2, 0\n\t" \
        "v_mov_b32 v3, 0\n\t" \
        "v_mov_b32 v4, 0\n\t" \
        "v_mov_b32 v5, 0\n\t" \
        "v_mov_b32 v6, 0\n\t" \
        "v_mov_b32 v7, 0\n\t" \
        "v_mov_b32 v8, 0\n\t" \
        "v_mov_b32 v9, 0\n\t" \
        "v_mov_b32 v10, 0\n\t" \
        "v_mov_b32 v11, 0\n\t" \
        "v_mov_b32 v12, 0\n\t" \
        "v_mov_b32 v13, 0\n\t" \
        "v_mov_b32 v14, 0\n\t" \
        "v_mov_b32 v15, 0\n\t" \
        "v_mov_b32 v16, 0\n\t" \
        "v_mov_b32 v17, 0\n\t" \
        "v_mov_b32 v18, 0\n\t" \
        "v_mov_b32 v19, 0\n\t" \
        "v_mov_b32 v20, 0\n\t" \
        "v_mov_b32 v21, 0\n\t" \
        "v_mov_b32 v22, 0\n\t" \
        "v_mov_b32 v23, 0\n\t" \
        "v_mov_b32 v24, 0\n\t" \
        "v_mov_b32 v25, 0\n\t" \
        "v_mov_b32 v26, 0\n\t" \
        "v_mov_b32 v27, 0\n\t" \
        "v_mov_b32 v28, 0\n\t" \
        "v_mov_b32 v29, 0\n\t" \
        "v_mov_b32 v30, 0\n\t" \
        "v_mov_b32 v31, 0\n\t" \
        "v_mov_b32 v32, 0\n\t" \
        "v_mov_b32 v33, 0\n\t" \
        "v_mov_b32 v34, 0\n\t" \
        "v_mov_b32 v35, 0\n\t" \
        "v_mov_b32 v36, 0\n\t" \
        "v_mov_b32 v37, 0\n\t" \
        "v_mov_b32 v38, 0\n\t" \
        "v_mov_b32 v39, 0\n\t" \
        "v_mov_b32 v40, 0\n\t" \
        "v_mov_b32 v41, 0\n\t" \
        "v_mov_b32 v42, 0\n\t" \
        "v_mov_b32 v43, 0\n\t" \
        "v_mov_b32 v44, 0\n\t" \
        "v_mov_b32 v45, 0\n\t" \
        "v_mov_b32 v46, 0\n\t" \
        "v_mov_b32 v47, 0\n\t" \
        "v_mov_b32 v48, 0\n\t" \
        "v_mov_b32 v49, 0\n\t" \
        "v_mov_b32 v50, 0\n\t" \
        "v_mov_b32 v51, 0\n\t" \
        "v_mov_b32 v52, 0\n\t" \
        "v_mov_b32 v53, 0\n\t" \
        "v_mov_b32 v54, 0\n\t" \
        "v_mov_b32 v55, 0\n\t" \
        "v_mov_b32 v56, 0\n\t" \
        "v_mov_b32 v57, 0\n\t" \
        "v_mov_b32 v58, 0\n\t" \
        "v_mov_b32 v59, 0\n\t" \
        "v_mov_b32 v60, 0\n\t" \
        "v_mov_b32 v61, 0\n\t" \
        "v_mov_b32 v62, 0\n\t" \
        "v_mov_b32 v63, 0\n\t" \
        "v_mov_b32 v64, 0\n\t" \
        "v_mov_b32 v65, 0\n\t" \
        "v_mov_b32 v66, 0\n\t" \
        "v_mov_b32 v67, 0\n\t" \
        "v_mov_b32 v68, 0\n\t" \
        "v_mov_b32 v69, 0\n\t" \
        "v_mov_b32 v70, 0\n\t" \
        "v_mov_b32 v71, 0\n\t" \
        "v_mov_b32 v72, 0\n\t" \
        "v_mov_b32 v73, 0\n\t" \
        "v_mov_b32 v74, 0\n\t" \
        "v_mov_b32 v75, 0\n\t" \
        "v_mov_b32 v76, 0\n\t" \
        "v_mov_b32 v77, 0\n\t" \
        "v_mov_b32 v78, 0\n\t" \
        "v_mov_b32 v79, 0\n\t" \
        "v_mov_b32 v80, 0\n\t" \
        "v_mov_b32 v81, 0\n\t" \
        "v_mov_b32 v82, 0\n\t" \
        "v_mov_b32 v83, 0\n\t" \
        "v_mov_b32 v84, 0\n\t" \
        "v_mov_b32 v85, 0\n\t" \
        "v_mov_b32 v86, 0\n\t" \
        "v_mov_b32 v87, 0\n\t" \
        "v_mov_b32 v88, 0\n\t" \
        "v_mov_b32 v89, 0\n\t" \
        "v_mov_b32 v90, 0\n\t" \
        "v_mov_b32 v91, 0\n\t" \
        "v_mov_b32 v92, 0\n\t" \
        "v_mov_b32 v93, 0\n\t" \
        "v_mov_b32 v94, 0\n\t" \
        "v_mov_b32 v95, 0\n\t" \
        "v_mov_b32 v96, 0\n\t" \
        "v_mov_b32 v97, 0\n\t" \
        "v_mov_b32 v98, 0\n\t" \
        "v_mov_b32 v99, 0\n\t" \
        "v_mov_b32 v100, 0\n\t" \
        "v_mov_b32 v101, 0\n\t" \
        "v_mov_b32 v102, 0\n\t" \
        "v_mov_b32 v103, 0\n\t" \
        "v_mov_b32 v104, 0\n\t" \
        "v_mov_b32 v105, 0\n\t" \
        "v_mov_b32 v106, 0\n\t" \
        "v_mov_b32 v107, 0\n\t" \
        "v_mov_b32 v108, 0\n\t" \
        "v_mov_b32 v109, 0\n\t" \
        "v_mov_b32 v110, 0\n\t" \
        "v_mov_b32 v111, 0\n\t" \
        "v_mov_b32 v112, 0\n\t" \
        "v_mov_b32 v113, 0\n\t" \
        "v_mov_b32 v114, 0\n\t" \
        "v_mov_b32 v115, 0\n\t" \
        "v_mov_b32 v116, 0\n\t" \
        "v_mov_b32 v117, 0\n\t" \
        "v_mov_b32 v118, 0\n\t" \
        "v_mov_b32 v119, 0\n\t" \
        "v_mov_b32 v120, 0\n\t" \
        "v_mov_b32 v121, 0\n\t" \
        "v_mov_b32 v122, 0\n\t" \
        "v_mov_b32 v123, 0\n\t" \
        "v_mov_b32 v124, 0\n\t" \
        "v_mov_b32 v125, 0\n\t" \
        "v_mov_b32 v126, 0\n\t" \
        "v_mov_b32 v127, 0\n\t" \
        "v_mov_b32 v128, 0\n\t" \
        "v_mov_b32 v129, 0\n\t" \
        "v_mov_b32 v130, 0\n\t" \
        "v_mov_b32 v131, 0\n\t" \
        "v_mov_b32 v132, 0\n\t" \
        "v_mov_b32 v133, 0\n\t" \
        "v_mov_b32 v134, 0\n\t" \
        "v_mov_b32 v135, 0\n\t" \
        "v_mov_b32 v136, 0\n\t" \
        "v_mov_b32 v137, 0\n\t" \
        "v_mov_b32 v138, 0\n\t" \
        "v_mov_b32 v139, 0\n\t" \
        "v_mov_b32 v140, 0\n\t" \
        "v_mov_b32 v141, 0\n\t" \
        "v_mov_b32 v142, 0\n\t" \
        "v_mov_b32 v143, 0\n\t" \
        "v_mov_b32 v144, 0\n\t" \
        "v_mov_b32 v145, 0\n\t" \
        "v_mov_b32 v146, 0\n\t" \
        "v_mov_b32 v147, 0\n\t" \
        "v_mov_b32 v148, 0\n\t" \
        "v_mov_b32 v149, 0\n\t" \
        "v_mov_b32 v150, 0\n\t" \
        "v_mov_b32 v151, 0\n\t" \
        "v_mov_b32 v152, 0\n\t" \
        "v_mov_b32 v153, 0\n\t" \
        "v_mov_b32 v154, 0\n\t" \
        "v_mov_b32 v155, 0\n\t" \
        "v_mov_b32 v156, 0\n\t" \
        "v_mov_b32 v157, 0\n\t" \
        "v_mov_b32 v158, 0\n\t" \
        "v_mov_b32 v159, 0\n\t" \
        "\nLP0%=:\n\t" \
        "ds_read_b32 %[vf0], %[ago]\n\t" \
        "s_waitcnt lgkmcnt(0)\n\t" \
        "v_cmp_ge_u32 vcc, %[vt], %[vf0]\n\t" \
        "s_cbranch_vccnz LP0%=\n\t" \
        "global_load_dwordx4 v[180:183], %[vl16], s[40:41] offset:0 sc0 sc1\n\t" \
        "global_load_dwordx4 v[184:187], %[vl16], s[40:41] offset:1024 sc0 sc1\n\t" \
        "global_load_dwordx4 v[188:191], %[vl16], s[40:41] offset:2048 sc0 sc1\n\t" \
        "global_load_dwordx4 v[192:195], %[vl16], s[40:41] offset:3072 sc0 sc1\n\t" \
        "s_add_u32 s40, s40, 16384\n\t" \
        "s_addc_u32 s41, s41, 0\n\t" \
        "global_load_dwordx4 v[196:199], %[vl16], s[40:41] offset:0 sc0 sc1\n\t" \
        "global_load_dwordx4 v[200:203], %[vl16], s[40:41] offset:1024 sc0 sc1\n\t" \
        "global_load_dwordx4 v[204:207], %[vl16], s[40:41] offset:2048 sc0 sc1\n\t" \
        "global_load_dwordx4 v[208:211], %[vl16], s[40:41] offset:3072 sc0 sc1\n\t" \
        "s_add_u32 s40, s40, 16384\n\t" \
        "s_addc_u32 s41, s41, 0\n\t" \
        "global_load_dwordx4 v[212:215], %[vl16], s[40:41] offset:0 sc0 sc1\n\t" \
        "global_load_dwordx4 v[216:219], %[vl16], s[40:41] offset:1024 sc0 sc1\n\t" \
        "global_load_dwordx4 v[220:223], %[vl16], s[40:41] offset:2048 sc0 sc1\n\t" \
        "global_load_dwordx4 v[224:227], %[vl16], s[40:41] offset:3072 sc0 sc1\n\t" \
        "s_add_u32 s40, s40, 16384\n\t" \
        "s_addc_u32 s41, s41, 0\n\t" \
        "ds_read_b128 v[160:163], %[vaw] offset:5120\n\t" \
        "ds_read_b128 v[164:167], %[vaw] offset:6144\n\t" \
        "ds_read_b128 v[168:171], %[vaw] offset:7168\n\t" \
        "ds_read_b128 v[172:175], %[vaw] offset:8192\n\t" \
        "ds_read_b128 v[176:179], %[vaw] offset:9216\n\t" \
        "s_waitcnt vmcnt(8) lgkmcnt(0)\n\t" \
        "v_mfma_f32_16x16x32_bf16 v[20:23], v[160:163], v[180:183], v[20:23]\n\t" \
        "v_mfma_f32_16x16x32_bf16 v[52:55], v[160:163], v[184:187], v[52:55]\n\t" \
        "v_mfma_f32_16x16x32_bf16 v[84:87], v[160:163], v[188:191], v[84:87]\n\t" \
        "v_mfma_f32_16x16x32_bf16 v[116:119], v[160:163], v[192:195], v[116:119]\n\t" \
        "v_mfma_f32_16x16x32_bf16 v[120:123], v[164:167], v[192:195], v[120:123]\n\t" \
        "v_mfma_f32_16x16x32_bf16 v[88:91], v[164:167], v[188:191], v[88:91]\n\t" \
        "v_mfma_f32_16x16x32_bf16 v[56:59], v[164:167], v[184:187], v[56:59]\n\t" \
        "v_mfma_f32_16x16x32_bf16 v[24:27], v[164:167], v[180:183], v[24:27]\n\t" \
        "ds_read_b128 v[160:163], %[vaw] offset:0\n\t" \
        "v_mfma_f32_16x16x32_bf16 v[28:31], v[168:171], v[180:183], v[28:31]\n\t" \
        "v_mfma_f32_16x16x32_bf16 v[60:63], v[168:171], v[184:187], v[60:63]\n\t" \
        "v_mfma_f32_16x16x32_bf16 v[92:95], v[168:171], v[188:191], v[92:95]\n\t" \
        "v_mfma_f32_16x16x32_bf16 v[124:127], v[168:171], v[192:195], v[124:127]\n\t" \
        "ds_read_b128 v[164:167], %[vaw] offset:10240\n\t" \
        "v_mfma_f32_16x16x32_bf16 v[152:155], v[172:175], v[192:195], v[152:155]\n\t" \
        "v_mfma_f32_16x16x32_bf16 v[144:147], v[172:175], v[188:191], v[144:147]\n\t" \
        "v_mfma_f32_16x16x32_bf16 v[136:139], v[172:175], v[184:187], v[136:139]\n\t" \
        "v_mfma_f32_16x16x32_bf16 v[128:131], v[172:175], v[180:183], v[128:131]\n\t" \
        "ds_read_b128 v[168:171], %[vaw] offset:1024\n\t" \
        "v_mfma_f32_16x16x32_bf16 v[132:135], v[176:179], v[180:183], v[132:135]\n\t" \
        "v_mfma_f32_16x16x32_bf16 v[140:143], v[176:179], v[184:187], v[140:143]\n\t" \
        "v_mfma_f32_16x16x32_bf16 v[148:151], v[176:179], v[188:191], v[148:151]\n\t" \
        "v_mfma_f32_16x16x32_bf16 v[156:159], v[176:179], v[192:195], v[156:159]\n\t" \
        "s_waitcnt lgkmcnt(0)\n\t" \
        "\nLM0%=:\n\t" \
        "s_waitcnt vmcnt(4) lgkmcnt(3)\n\t" \
        "v_mfma_f32_16x16x32_bf16 v[0:3], v[160:163], v[180:183], v[0:3]\n\t" \
        "ds_read_b128 v[172:175], %[vaw] offset:11264\n\t" \
        "s_waitcnt lgkmcnt(3)\n\t" \
        "v_mfma_f32_16x16x32_bf16 v[0:3], v[164:167], v[196:199], v[0:3]\n\t" \
        "v_mfma_f32_16x16x32_bf16 v[32:35], v[164:167], v[200:203], v[32:35]\n\t" \
        "ds_read_b128 v[176:179], %[vaw] offset:2048\n\t" \
        "v_mfma_f32_16x16x32_bf16 v[32:35], v[160:163], v[184:187], v[32:35]\n\t" \
        "s_add_u32 %[sb], %[sb], 40960\n\t" \
        "s_cmp_eq_u32 %[sb], 122880\n\t" \
        "v_mfma_f32_16x16x32_bf16 v[64:67], v[160:163], v[188:191], v[64:67]\n\t" \
        "v_add_u32 %[vt], 1, %[vt]\n\t" \
        "v_mfma_f32_16x16x32_bf16 v[64:67], v[164:167], v[204:207], v[64:67]\n\t" \
        "s_cselect_b32 %[sb], 0, %[sb]\n\t" \
        "v_mfma_f32_16x16x32_bf16 v[96:99], v[164:167], v[208:211], v[96:99]\n\t" \
        "v_add_u32 %[vax], %[sb], %[vlw]\n\t" \
        "v_mfma_f32_16x16x32_bf16 v[96:99], v[160:163], v[192:195], v[96:99]\n\t" \
        "s_waitcnt lgkmcnt(3)\n\t" \
        "v_mfma_f32_16x16x32_bf16 v[100:103], v[168:171], v[192:195], v[100:103]\n\t" \
        "ds_read_b128 v[160:163], %[vaw] offset:12288\n\t" \
        "s_waitcnt lgkmcnt(2)\n\t" \
        "v_mfma_f32_16x16x32_bf16 v[100:103], v[172:175], v[208:211], v[100:103]\n\t" \
        "v_mfma_f32_16x16x32_bf16 v[68:71], v[172:175], v[204:207], v[68:71]\n\t" \
        "ds_read_b128 v[164:167], %[vaw] offset:3072\n\t" \
        "v_mfma_f32_16x16x32_bf16 v[68:71], v[168:171], v[188:191], v[68:71]\n\t" \
        "v_mfma_f32_16x16x32_bf16 v[36:39], v[168:171], v[184:187], v[36:39]\n\t" \
        "ds_read_b32 %[vf0], %[ago]\n\t" \
        "v_mfma_f32_16x16x32_bf16 v[36:39], v[172:175], v[200:203], v[36:39]\n\t" \
        "v_mfma_f32_16x16x32_bf16 v[4:7], v[172:175], v[196:199], v[4:7]\n\t" \
        "v_mfma_f32_16x16x32_bf16 v[4:7], v[168:171], v[180:183], v[4:7]\n\t" \
        "s_waitcnt lgkmcnt(3)\n\t" \
        "v_mfma_f32_16x16x32_bf16 v[8:11], v[176:179], v[180:183], v[8:11]\n\t" \
        "ds_read_b128 v[168:171], %[vaw] offset:13312\n\t" \
        "s_waitcnt lgkmcnt(3)\n\t" \
        "v_mfma_f32_16x16x32_bf16 v[8:11], v[160:163], v[196:199], v[8:11]\n\t" \
        "v_mfma_f32_16x16x32_bf16 v[40:43], v[160:163], v[200:203], v[40:43]\n\t" \
        "ds_read_b128 v[172:175], %[vaw] offset:4096\n\t" \
        "v_mfma_f32_16x16x32_bf16 v[40:43], v[176:179], v[184:187], v[40:43]\n\t" \
        "v_mfma_f32_16x16x32_bf16 v[72:75], v[176:179], v[188:191], v[72:75]\n\t" \
        "s_waitcnt lgkmcnt(2)\n\t" \
        "v_mfma_f32_16x16x32_bf16 v[72:75], v[160:163], v[204:207], v[72:75]\n\t" \
        "v_cmp_ge_u32 vcc, %[vt], %[vf0]\n\t" \
        "v_mfma_f32_16x16x32_bf16 v[104:107], v[160:163], v[208:211], v[104:107]\n\t" \
        "v_mfma_f32_16x16x32_bf16 v[104:107], v[176:179], v[192:195], v[104:107]\n\t" \
        "s_cbranch_vccnz LSLM0%=\n\t" \
        "v_mfma_f32_16x16x32_bf16 v[108:111], v[164:167], v[192:195], v[108:111]\n\t" \
        "ds_read_b128 v[176:179], %[vaw] offset:14336\n\t" \
        "s_waitcnt lgkmcnt(2)\n\t" \
        "v_mfma_f32_16x16x32_bf16 v[108:111], v[168:171], v[208:211], v[108:111]\n\t" \
        "global_load_dwordx4 v[228:231], %[vl16], s[40:41] offset:0 sc0 sc1\n\t" \
        "v_mfma_f32_16x16x32_bf16 v[76:79], v[168:171], v[204:207], v[76:79]\n\t" \
        "ds_read_b128 v[160:163], %[vaw] offset:15360\n\t" \
        "v_mfma_f32_16x16x32_bf16 v[76:79], v[164:167], v[188:191], v[76:79]\n\t" \
        "global_load_dwordx4 v[232:235], %[vl16], s[40:41] offset:1024 sc0 sc1\n\t" \
        "v_mfma_f32_16x16x32_bf16 v[44:47], v[164:167], v[184:187], v[44:47]\n\t" \
        "global_load_dwordx4 v[236:239], %[vl16], s[40:41] offset:2048 sc0 sc1\n\t" \
        "v_mfma_f32_16x16x32_bf16 v[44:47], v[168:171], v[200:203], v[44:47]\n\t" \
        "global_load_dwordx4 v[240:243], %[vl16], s[40:41] offset:3072 sc0 sc1\n\t" \
        "v_mfma_f32_16x16x32_bf16 v[12:15], v[168:171], v[196:199], v[12:15]\n\t" \
        "s_add_u32 s40, s40, 16384\n\t" \
        "s_addc_u32 s41, s41, 0\n\t" \
        "v_mfma_f32_16x16x32_bf16 v[12:15], v[164:167], v[180:183], v[12:15]\n\t" \
        "s_waitcnt lgkmcnt(2)\n\t" \
        "v_mfma_f32_16x16x32_bf16 v[16:19], v[172:175], v[180:183], v[16:19]\n\t" \
        "ds_read_b128 v[164:167], %[vaw] offset:25600\n\t" \
        "s_waitcnt lgkmcnt(2)\n\t" \
        "v_mfma_f32_16x16x32_bf16 v[16:19], v[176:179], v[196:199], v[16:19]\n\t" \
        "v_mfma_f32_16x16x32_bf16 v[48:51], v[176:179], v[200:203], v[48:51]\n\t" \
        "ds_read_b128 v[168:171], %[vaw] offset:16384\n\t" \
        "v_mfma_f32_16x16x32_bf16 v[48:51], v[172:175], v[184:187], v[48:51]\n\t" \
        "ds_write_b32 %[afree], %[vt]\n\t" \
        "v_mfma_f32_16x16x32_bf16 v[80:83], v[172:175], v[188:191], v[80:83]\n\t" \
        "v_mfma_f32_16x16x32_bf16 v[80:83], v[176:179], v[204:207], v[80:83]\n\t" \
        "v_mfma_f32_16x16x32_bf16 v[112:115], v[176:179], v[208:211], v[112:115]\n\t" \
        "v_mfma_f32_16x16x32_bf16 v[112:115], v[172:175], v[192:195], v[112:115]\n\t" \
        "\nLM1%=:\n\t" \
        "s_waitcnt vmcnt(4) lgkmcnt(3)\n\t" \
        "v_mfma_f32_16x16x32_bf16 v[20:23], v[160:163], v[196:199], v[20:23]\n\t" \
        "ds_read_b128 v[172:175], %[vaw] offset:26624\n\t" \
        "s_waitcnt lgkmcnt(3)\n\t" \
        "v_mfma_f32_16x16x32_bf16 v[20:23], v[164:167], v[212:215], v[20:23]\n\t" \
        "v_mfma_f32_16x16x32_bf16 v[52:55], v[164:167], v[216:219], v[52:55]\n\t" \
        "ds_read_b128 v[176:179], %[vaw] offset:17408\n\t" \
        "v_mfma_f32_16x16x32_bf16 v[52:55], v[160:163], v[200:203], v[52:55]\n\t" \
        "v_mfma_f32_16x16x32_bf16 v[84:87], v[160:163], v[204:207], v[84:87]\n\t" \
        "v_add_u32 %[vt], 1, %[vt]\n\t" \
        "v_mfma_f32_16x16x32_bf16 v[84:87], v[164:167], v[220:223], v[84:87]\n\t" \
        "v_mfma_f32_16x16x32_bf16 v[116:119], v[164:167], v[224:227], v[116:119]\n\t" \
        "v_mfma_f32_16x16x32_bf16 v[116:119], v[160:163], v[208:211], v[116:119]\n\t" \
        "s_waitcnt lgkmcnt(3)\n\t" \
        "v_mfma_f32_16x16x32_bf16 v[120:123], v[168:171], v[208:211], v[120:123]\n\t" \
        "ds_read_b128 v[160:163], %[vaw] offset:27648\n\t" \
        "s_waitcnt lgkmcnt(2)\n\t" \
        "v_mfma_f32_16x16x32_bf16 v[120:123], v[172:175], v[224:227], v[120:123]\n\t" \
        "v_mfma_f32_16x16x32_bf16 v[88:91], v[172:175], v[220:223], v[88:91]\n\t" \
        "ds_read_b128 v[164:167], %[vaw] offset:18432\n\t" \
        "v_mfma_f32_16x16x32_bf16 v[88:91], v[168:171], v[204:207], v[88:91]\n\t" \
        "v_mfma_f32_16x16x32_bf16 v[56:59], v[168:171], v[200:203], v[56:59]\n\t" \
        "ds_read_b32 %[vf0], %[ago]\n\t" \
        "v_mfma_f32_16x16x32_bf16 v[56:59], v[172:175], v[216:219], v[56:59]\n\t" \
        "v_mfma_f32_16x16x32_bf16 v[24:27], v[172:175], v[212:215], v[24:27]\n\t" \
        "v_mfma_f32_16x16x32_bf16 v[24:27], v[168:171], v[196:199], v[24:27]\n\t" \
        "s_waitcnt lgkmcnt(3)\n\t" \
        "v_mfma_f32_16x16x32_bf16 v[28:31], v[176:179], v[196:199], v[28:31]\n\t" \
        "ds_read_b128 v[168:171], %[vaw] offset:28672\n\t" \
        "s_waitcnt lgkmcnt(3)\n\t" \
        "v_mfma_f32_16x16x32_bf16 v[28:31], v[160:163], v[212:215], v[28:31]\n\t" \
        "v_mfma_f32_16x16x32_bf16 v[60:63], v[160:163], v[216:219], v[60:63]\n\t" \
        "ds_read_b128 v[172:175], %[vaw] offset:19456\n\t" \
        "v_mfma_f32_16x16x32_bf16 v[60:63], v[176:179], v[200:203], v[60:63]\n\t" \
        "v_mfma_f32_16x16x32_bf16 v[92:95], v[176:179], v[204:207], v[92:95]\n\t" \
        "s_waitcnt lgkmcnt(2)\n\t" \
        "v_mfma_f32_16x16x32_bf16 v[92:95], v[160:163], v[220:223], v[92:95]\n\t" \
        "v_cmp_ge_u32 vcc, %[vt], %[vf0]\n\t" \
        "v_mfma_f32_16x16x32_bf16 v[124:127], v[160:163], v[224:227], v[124:127]\n\t" \
        "v_mfma_f32_16x16x32_bf16 v[124:127], v[176:179], v[208:211], v[124:127]\n\t" \
        "s_cbranch_vccnz LSLM1%=\n\t" \
        "v_mfma_f32_16x16x32_bf16 v[152:155], v[164:167], v[208:211], v[152:155]\n\t" \
        "ds_read_b128 v[176:179], %[vaw] offset:29696\n\t" \
        "s_waitcnt lgkmcnt(2)\n\t" \
        "v_mfma_f32_16x16x32_bf16 v[152:155], v[168:171], v[224:227], v[152:155]\n\t" \
        "global_load_dwordx4 v[180:183], %[vl16], s[40:41] offset:0 sc0 sc1\n\t" \
        "v_mfma_f32_16x16x32_bf16 v[144:147], v[168:171], v[220:223], v[144:147]\n\t" \
        "ds_read_b128 v[160:163], %[vaw] offset:20480\n\t" \
        "v_mfma_f32_16x16x32_bf16 v[144:147], v[164:167], v[204:207], v[144:147]\n\t" \
        "global_load_dwordx4 v[184:187], %[vl16], s[40:41] offset:1024 sc0 sc1\n\t" \
        "v_mfma_f32_16x16x32_bf16 v[136:139], v[164:167], v[200:203], v[136:139]\n\t" \
        "global_load_dwordx4 v[188:191], %[vl16], s[40:41] offset:2048 sc0 sc1\n\t" \
        "v_mfma_f32_16x16x32_bf16 v[136:139], v[168:171], v[216:219], v[136:139]\n\t" \
        "global_load_dwordx4 v[192:195], %[vl16], s[40:41] offset:3072 sc0 sc1\n\t" \
        "v_mfma_f32_16x16x32_bf16 v[128:131], v[168:171], v[212:215], v[128:131]\n\t" \
        "s_add_u32 s40, s40, 16384\n\t" \
        "s_addc_u32 s41, s41, 0\n\t" \
        "v_mfma_f32_16x16x32_bf16 v[128:131], v[164:167], v[196:199], v[128:131]\n\t" \
        "s_waitcnt lgkmcnt(2)\n\t" \
        "v_mfma_f32_16x16x32_bf16 v[132:135], v[172:175], v[196:199], v[132:135]\n\t" \
        "ds_read_b128 v[164:167], %[vaw] offset:30720\n\t" \
        "s_waitcnt lgkmcnt(2)\n\t" \
        "v_mfma_f32_16x16x32_bf16 v[132:135], v[176:179], v[212:215], v[132:135]\n\t" \
        "v_mfma_f32_16x16x32_bf16 v[140:143], v[176:179], v[216:219], v[140:143]\n\t" \
        "ds_read_b128 v[168:171], %[vaw] offset:21504\n\t" \
        "v_mfma_f32_16x16x32_bf16 v[140:143], v[172:175], v[200:203], v[140:143]\n\t" \
        "ds_write_b32 %[afree], %[vt]\n\t" \
        "v_mfma_f32_16x16x32_bf16 v[148:151], v[172:175], v[204:207], v[148:151]\n\t" \
        "v_mfma_f32_16x16x32_bf16 v[148:151], v[176:179], v[220:223], v[148:151]\n\t" \
        "v_mfma_f32_16x16x32_bf16 v[156:159], v[176:179], v[224:227], v[156:159]\n\t" \
        "v_mfma_f32_16x16x32_bf16 v[156:159], v[172:175], v[208:211], v[156:159]\n\t" \
        "\nLM2%=:\n\t" \
        "s_waitcnt vmcnt(4) lgkmcnt(3)\n\t" \
        "v_mfma_f32_16x16x32_bf16 v[0:3], v[160:163], v[212:215], v[0:3]\n\t" \
        "ds_read_b128 v[172:175], %[vaw] offset:31744\n\t" \
        "s_waitcnt lgkmcnt(3)\n\t" \
        "v_mfma_f32_16x16x32_bf16 v[0:3], v[164:167], v[228:231], v[0:3]\n\t" \
        "v_mfma_f32_16x16x32_bf16 v[32:35], v[164:167], v[232:235], v[32:35]\n\t" \
        "ds_read_b128 v[176:179], %[vaw] offset:22528\n\t" \
        "v_mfma_f32_16x16x32_bf16 v[32:35], v[160:163], v[216:219], v[32:35]\n\t" \
        "v_mfma_f32_16x16x32_bf16 v[64:67], v[160:163], v[220:223], v[64:67]\n\t" \
        "v_add_u32 %[vt], 1, %[vt]\n\t" \
        "v_mfma_f32_16x16x32_bf16 v[64:67], v[164:167], v[236:239], v[64:67]\n\t" \
        "v_mfma_f32_16x16x32_bf16 v[96:99], v[164:167], v[240:243], v[96:99]\n\t" \
        "v_mfma_f32_16x16x32_bf16 v[96:99], v[160:163], v[224:227], v[96:99]\n\t" \
        "s_waitcnt lgkmcnt(3)\n\t" \
        "v_mfma_f32_16x16x32_bf16 v[100:103], v[168:171], v[224:227], v[100:103]\n\t" \
        "ds_read_b128 v[160:163], %[vaw] offset:32768\n\t" \
        "s_waitcnt lgkmcnt(2)\n\t" \
        "v_mfma_f32_16x16x32_bf16 v[100:103], v[172:175], v[240:243], v[100:103]\n\t" \
        "v_mfma_f32_16x16x32_bf16 v[68:71], v[172:175], v[236:239], v[68:71]\n\t" \
        "ds_read_b128 v[164:167], %[vaw] offset:23552\n\t" \
        "v_mfma_f32_16x16x32_bf16 v[68:71], v[168:171], v[220:223], v[68:71]\n\t" \
        "v_mfma_f32_16x16x32_bf16 v[36:39], v[168:171], v[216:219], v[36:39]\n\t" \
        "ds_read_b32 %[vf0], %[ago]\n\t" \
        "v_mfma_f32_16x16x32_bf16 v[36:39], v[172:175], v[232:235], v[36:39]\n\t" \
        "v_mfma_f32_16x16x32_bf16 v[4:7], v[172:175], v[228:231], v[4:7]\n\t" \
        "v_mfma_f32_16x16x32_bf16 v[4:7], v[168:171], v[212:215], v[4:7]\n\t" \
        "s_waitcnt lgkmcnt(3)\n\t" \
        "v_mfma_f32_16x16x32_bf16 v[8:11], v[176:179], v[212:215], v[8:11]\n\t" \
        "ds_read_b128 v[168:171], %[vaw] offset:33792\n\t" \
        "s_waitcnt lgkmcnt(3)\n\t" \
        "v_mfma_f32_16x16x32_bf16 v[8:11], v[160:163], v[228:231], v[8:11]\n\t" \
        "v_mfma_f32_16x16x32_bf16 v[40:43], v[160:163], v[232:235], v[40:43]\n\t" \
        "ds_read_b128 v[172:175], %[vaw] offset:24576\n\t" \
        "v_mfma_f32_16x16x32_bf16 v[40:43], v[176:179], v[216:219], v[40:43]\n\t" \
        "v_mfma_f32_16x16x32_bf16 v[72:75], v[176:179], v[220:223], v[72:75]\n\t" \
        "s_waitcnt lgkmcnt(2)\n\t" \
        "v_mfma_f32_16x16x32_bf16 v[72:75], v[160:163], v[236:239], v[72:75]\n\t" \
        "v_cmp_ge_u32 vcc, %[vt], %[vf0]\n\t" \
        "v_mfma_f32_16x16x32_bf16 v[104:107], v[160:163], v[240:243], v[104:107]\n\t" \
        "v_mfma_f32_16x16x32_bf16 v[104:107], v[176:179], v[224:227], v[104:107]\n\t" \
        "s_cbranch_vccnz LSLM2%=\n\t" \
        "v_mfma_f32_16x16x32_bf16 v[108:111], v[164:167], v[224:227], v[108:111]\n\t" \
        "ds_read_b128 v[176:179], %[vaw] offset:34816\n\t" \
        "s_waitcnt lgkmcnt(2)\n\t" \
        "v_mfma_f32_16x16x32_bf16 v[108:111], v[168:171], v[240:243], v[108:111]\n\t" \
        "global_load_dwordx4 v[196:199], %[vl16], s[40:41] offset:0 sc0 sc1\n\t" \
        "v_mfma_f32_16x16x32_bf16 v[76:79], v[168:171], v[236:239], v[76:79]\n\t" \
        "ds_read_b128 v[160:163], %[vaw] offset:35840\n\t" \
        "v_mfma_f32_16x16x32_bf16 v[76:79], v[164:167], v[220:223], v[76:79]\n\t" \
        "global_load_dwordx4 v[200:203], %[vl16], s[40:41] offset:1024 sc0 sc1\n\t" \
        "v_mfma_f32_16x16x32_bf16 v[44:47], v[164:167], v[216:219], v[44:47]\n\t" \
        "global_load_dwordx4 v[204:207], %[vl16], s[40:41] offset:2048 sc0 sc1\n\t" \
        "v_mfma_f32_16x16x32_bf16 v[44:47], v[168:171], v[232:235], v[44:47]\n\t" \
        "global_load_dwordx4 v[208:211], %[vl16], s[40:41] offset:3072 sc0 sc1\n\t" \
        "v_mfma_f32_16x16x32_bf16 v[12:15], v[168:171], v[228:231], v[12:15]\n\t" \
        "s_add_u32 s40, s40, 16384\n\t" \
        "s_addc_u32 s41, s41, 0\n\t" \
        "v_mfma_f32_16x16x32_bf16 v[12:15], v[164:167], v[212:215], v[12:15]\n\t" \
        "s_waitcnt lgkmcnt(2)\n\t" \
        "v_mfma_f32_16x16x32_bf16 v[16:19], v[172:175], v[212:215], v[16:19]\n\t" \
        "ds_read_b128 v[164:167], %[vax] offset:5120\n\t" \
        "s_waitcnt lgkmcnt(2)\n\t" \
        "v_mfma_f32_16x16x32_bf16 v[16:19], v[176:179], v[228:231], v[16:19]\n\t" \
        "v_mfma_f32_16x16x32_bf16 v[48:51], v[176:179], v[232:235], v[48:51]\n\t" \
        "ds_read_b128 v[168:171], %[vaw] offset:36864\n\t" \
        "v_mfma_f32_16x16x32_bf16 v[48:51], v[172:175], v[216:219], v[48:51]\n\t" \
        "ds_write_b32 %[afree], %[vt]\n\t" \
        "v_mfma_f32_16x16x32_bf16 v[80:83], v[172:175], v[220:223], v[80:83]\n\t" \
        "v_mfma_f32_16x16x32_bf16 v[80:83], v[176:179], v[236:239], v[80:83]\n\t" \
        "v_mfma_f32_16x16x32_bf16 v[112:115], v[176:179], v[240:243], v[112:115]\n\t" \
        "v_mfma_f32_16x16x32_bf16 v[112:115], v[172:175], v[224:227], v[112:115]\n\t" \
        "\nLM3%=:\n\t" \
        "s_waitcnt vmcnt(4) lgkmcnt(3)\n\t" \
        "v_mfma_f32_16x16x32_bf16 v[20:23], v[160:163], v[228:231], v[20:23]\n\t" \
        "ds_read_b128 v[172:175], %[vax] offset:6144\n\t" \
        "s_waitcnt lgkmcnt(3)\n\t" \
        "v_mfma_f32_16x16x32_bf16 v[20:23], v[164:167], v[180:183], v[20:23]\n\t" \
        "v_mfma_f32_16x16x32_bf16 v[52:55], v[164:167], v[184:187], v[52:55]\n\t" \
        "ds_read_b128 v[176:179], %[vaw] offset:37888\n\t" \
        "v_mfma_f32_16x16x32_bf16 v[52:55], v[160:163], v[232:235], v[52:55]\n\t" \
        "v_mfma_f32_16x16x32_bf16 v[84:87], v[160:163], v[236:239], v[84:87]\n\t" \
        "v_add_u32 %[vt], 1, %[vt]\n\t" \
        "v_mfma_f32_16x16x32_bf16 v[84:87], v[164:167], v[188:191], v[84:87]\n\t" \
        "v_mfma_f32_16x16x32_bf16 v[116:119], v[164:167], v[192:195], v[116:119]\n\t" \
        "v_mfma_f32_16x16x32_bf16 v[116:119], v[160:163], v[240:243], v[116:119]\n\t" \
        "s_waitcnt lgkmcnt(3)\n\t" \
        "v_mfma_f32_16x16x32_bf16 v[120:123], v[168:171], v[240:243], v[120:123]\n\t" \
        "ds_read_b128 v[160:163], %[vax] offset:7168\n\t" \
        "s_waitcnt lgkmcnt(2)\n\t" \
        "v_mfma_f32_16x16x32_bf16 v[120:123], v[172:175], v[192:195], v[120:123]\n\t" \
        "v_mfma_f32_16x16x32_bf16 v[88:91], v[172:175], v[188:191], v[88:91]\n\t" \
        "ds_read_b128 v[164:167], %[vaw] offset:38912\n\t" \
        "v_mfma_f32_16x16x32_bf16 v[88:91], v[168:171], v[236:239], v[88:91]\n\t" \
        "v_mfma_f32_16x16x32_bf16 v[56:59], v[168:171], v[232:235], v[56:59]\n\t" \
        "ds_read_b32 %[vf0], %[ago]\n\t" \
        "v_mfma_f32_16x16x32_bf16 v[56:59], v[172:175], v[184:187], v[56:59]\n\t" \
        "v_mfma_f32_16x16x32_bf16 v[24:27], v[172:175], v[180:183], v[24:27]\n\t" \
        "v_mfma_f32_16x16x32_bf16 v[24:27], v[168:171], v[228:231], v[24:27]\n\t" \
        "s_waitcnt lgkmcnt(3)\n\t" \
        "v_mfma_f32_16x16x32_bf16 v[28:31], v[176:179], v[228:231], v[28:31]\n\t" \
        "ds_read_b128 v[168:171], %[vax] offset:8192\n\t" \
        "s_waitcnt lgkmcnt(3)\n\t" \
        "v_mfma_f32_16x16x32_bf16 v[28:31], v[160:163], v[180:183], v[28:31]\n\t" \
        "v_mfma_f32_16x16x32_bf16 v[60:63], v[160:163], v[184:187], v[60:63]\n\t" \
        "ds_read_b128 v[172:175], %[vaw] offset:39936\n\t" \
        "v_mfma_f32_16x16x32_bf16 v[60:63], v[176:179], v[232:235], v[60:63]\n\t" \
        "v_mfma_f32_16x16x32_bf16 v[92:95], v[176:179], v[236:239], v[92:95]\n\t" \
        "s_waitcnt lgkmcnt(2)\n\t" \
        "v_mfma_f32_16x16x32_bf16 v[92:95], v[160:163], v[188:191], v[92:95]\n\t" \
        "v_cmp_ge_u32 vcc, %[vt], %[vf0]\n\t" \
        "v_mfma_f32_16x16x32_bf16 v[124:127], v[160:163], v[192:195], v[124:127]\n\t" \
        "v_mfma_f32_16x16x32_bf16 v[124:127], v[176:179], v[240:243], v[124:127]\n\t" \
        "s_cbranch_vccnz LSLM3%=\n\t" \
        "v_mfma_f32_16x16x32_bf16 v[152:155], v[164:167], v[240:243], v[152:155]\n\t" \
        "ds_read_b128 v[176:179], %[vax] offset:9216\n\t" \
        "s_waitcnt lgkmcnt(2)\n\t" \
        "v_mfma_f32_16x16x32_bf16 v[152:155], v[168:171], v[192:195], v[152:155]\n\t" \
        "global_load_dwordx4 v[212:215], %[vl16], s[40:41] offset:0 sc0 sc1\n\t" \
        "v_mfma_f32_16x16x32_bf16 v[144:147], v[168:171], v[188:191], v[144:147]\n\t" \
        "ds_read_b128 v[160:163], %[vax] offset:0\n\t" \
        "v_mfma_f32_16x16x32_bf16 v[144:147], v[164:167], v[236:239], v[144:147]\n\t" \
        "global_load_dwordx4 v[216:219], %[vl16], s[40:41] offset:1024 sc0 sc1\n\t" \
        "v_mfma_f32_16x16x32_bf16 v[136:139], v[164:167], v[232:235], v[136:139]\n\t" \
        "global_load_dwordx4 v[220:223], %[vl16], s[40:41] offset:2048 sc0 sc1\n\t" \
        "v_mfma_f32_16x16x32_bf16 v[136:139], v[168:171], v[184:187], v[136:139]\n\t" \
        "global_load_dwordx4 v[224:227], %[vl16], s[40:41] offset:3072 sc0 sc1\n\t" \
        "v_mfma_f32_16x16x32_bf16 v[128:131], v[168:171], v[180:183], v[128:131]\n\t" \
        "s_add_u32 s40, s40, 16384\n\t" \
        "s_addc_u32 s41, s41, 0\n\t" \
        "v_mfma_f32_16x16x32_bf16 v[128:131], v[164:167], v[228:231], v[128:131]\n\t" \
        "s_waitcnt lgkmcnt(2)\n\t" \
        "v_mfma_f32_16x16x32_bf16 v[132:135], v[172:175], v[228:231], v[132:135]\n\t" \
        "ds_read_b128 v[164:167], %[vax] offset:10240\n\t" \
        "s_waitcnt lgkmcnt(2)\n\t" \
        "v_mfma_f32_16x16x32_bf16 v[132:135], v[176:179], v[180:183], v[132:135]\n\t" \
        "v_mfma_f32_16x16x32_bf16 v[140:143], v[176:179], v[184:187], v[140:143]\n\t" \
        "ds_read_b128 v[168:171], %[vax] offset:1024\n\t" \
        "v_mfma_f32_16x16x32_bf16 v[140:143], v[172:175], v[232:235], v[140:143]\n\t" \
        "ds_write_b32 %[afree], %[vt]\n\t" \
        "v_mfma_f32_16x16x32_bf16 v[148:151], v[172:175], v[236:239], v[148:151]\n\t" \
        "v_mov_b32 %[vaw], %[vax]\n\t" \
        "v_mfma_f32_16x16x32_bf16 v[148:151], v[176:179], v[188:191], v[148:151]\n\t" \
        "s_sub_u32 %[ss], %[ss], 1\n\t" \
        "v_mfma_f32_16x16x32_bf16 v[156:159], v[176:179], v[192:195], v[156:159]\n\t" \
        "s_cmp_lg_u32 %[ss], 0\n\t" \
        "v_mfma_f32_16x16x32_bf16 v[156:159], v[172:175], v[240:243], v[156:159]\n\t" \
        "s_cbranch_scc1 LM0%=\n\t" \
        "\nLT0%=:\n\t" \
        "s_waitcnt vmcnt(4) lgkmcnt(3)\n\t" \
        "v_mfma_f32_16x16x32_bf16 v[0:3], v[160:163], v[180:183], v[0:3]\n\t" \
        "ds_read_b128 v[172:175], %[vaw] offset:11264\n\t" \
        "s_waitcnt lgkmcnt(3)\n\t" \
        "v_mfma_f32_16x16x32_bf16 v[0:3], v[164:167], v[196:199], v[0:3]\n\t" \
        "v_mfma_f32_16x16x32_bf16 v[32:35], v[164:167], v[200:203], v[32:35]\n\t" \
        "ds_read_b128 v[176:179], %[vaw] offset:2048\n\t" \
        "v_mfma_f32_16x16x32_bf16 v[32:35], v[160:163], v[184:187], v[32:35]\n\t" \
        "s_add_u32 %[sb], %[sb], 40960\n\t" \
        "s_cmp_eq_u32 %[sb], 122880\n\t" \
        "v_mfma_f32_16x16x32_bf16 v[64:67], v[160:163], v[188:191], v[64:67]\n\t" \
        "v_add_u32 %[vt], 1, %[vt]\n\t" \
        "v_mfma_f32_16x16x32_bf16 v[64:67], v[164:167], v[204:207], v[64:67]\n\t" \
        "s_cselect_b32 %[sb], 0, %[sb]\n\t" \
        "v_mfma_f32_16x16x32_bf16 v[96:99], v[164:167], v[208:211], v[96:99]\n\t" \
        "v_add_u32 %[vax], %[sb], %[vlw]\n\t" \
        "v_mfma_f32_16x16x32_bf16 v[96:99], v[160:163], v[192:195], v[96:99]\n\t" \
        "s_waitcnt lgkmcnt(3)\n\t" \
        "v_mfma_f32_16x16x32_bf16 v[100:103], v[168:171], v[192:195], v[100:103]\n\t" \
        "ds_read_b128 v[160:163], %[vaw] offset:12288\n\t" \
        "s_waitcnt lgkmcnt(2)\n\t" \
        "v_mfma_f32_16x16x32_bf16 v[100:103], v[172:175], v[208:211], v[100:103]\n\t" \
        "v_mfma_f32_16x16x32_bf16 v[68:71], v[172:175], v[204:207], v[68:71]\n\t" \
        "ds_read_b128 v[164:167], %[vaw] offset:3072\n\t" \
        "v_mfma_f32_16x16x32_bf16 v[68:71], v[168:171], v[188:191], v[68:71]\n\t" \
        "v_mfma_f32_16x16x32_bf16 v[36:39], v[168:171], v[184:187], v[36:39]\n\t" \
        "ds_read_b32 %[vf0], %[ago]\n\t" \
        "v_mfma_f32_16x16x32_bf16 v[36:39], v[172:175], v[200:203], v[36:39]\n\t" \
        "v_mfma_f32_16x16x32_bf16 v[4:7], v[172:175], v[196:199], v[4:7]\n\t" \
        "v_mfma_f32_16x16x32_bf16 v[4:7], v[168:171], v[180:183], v[4:7]\n\t" \
        "s_waitcnt lgkmcnt(3)\n\t" \
        "v_mfma_f32_16x16x32_bf16 v[8:11], v[176:179], v[180:183], v[8:11]\n\t" \
        "ds_read_b128 v[168:171], %[vaw] offset:13312\n\t" \
        "s_waitcnt lgkmcnt(3)\n\t" \
        "v_mfma_f32_16x16x32_bf16 v[8:11], v[160:163], v[196:199], v[8:11]\n\t" \
        "v_mfma_f32_16x16x32_bf16 v[40:43], v[160:163], v[200:203], v[40:43]\n\t" \
        "ds_read_b128 v[172:175], %[vaw] offset:4096\n\t" \
        "v_mfma_f32_16x16x32_bf16 v[40:43], v[176:179], v[184:187], v[40:43]\n\t" \
        "v_mfma_f32_16x16x32_bf16 v[72:75], v[176:179], v[188:191], v[72:75]\n\t" \
        "s_waitcnt lgkmcnt(2)\n\t" \
        "v_mfma_f32_16x16x32_bf16 v[72:75], v[160:163], v[204:207], v[72:75]\n\t" \
        "v_cmp_ge_u32 vcc, %[vt], %[vf0]\n\t" \
        "v_mfma_f32_16x16x32_bf16 v[104:107], v[160:163], v[208:211], v[104:107]\n\t" \
        "v_mfma_f32_16x16x32_bf16 v[104:107], v[176:179], v[192:195], v[104:107]\n\t" \
        "s_cbranch_vccnz LSLT0%=\n\t" \
        "v_mfma_f32_16x16x32_bf16 v[108:111], v[164:167], v[192:195], v[108:111]\n\t" \
        "ds_read_b128 v[176:179], %[vaw] offset:14336\n\t" \
        "s_waitcnt lgkmcnt(2)\n\t" \
        "v_mfma_f32_16x16x32_bf16 v[108:111], v[168:171], v[208:211], v[108:111]\n\t" \
        "global_load_dwordx4 v[228:231], %[vl16], s[40:41] offset:0 sc0 sc1\n\t" \
        "v_mfma_f32_16x16x32_bf16 v[76:79], v[168:171], v[204:207], v[76:79]\n\t" \
        "ds_read_b128 v[160:163], %[vaw] offset:15360\n\t" \
        "v_mfma_f32_16x16x32_bf16 v[76:79], v[164:167], v[188:191], v[76:79]\n\t" \
        "global_load_dwordx4 v[232:235], %[vl16], s[40:41] offset:1024 sc0 sc1\n\t" \
        "v_mfma_f32_16x16x32_bf16 v[44:47], v[164:167], v[184:187], v[44:47]\n\t" \
        "global_load_dwordx4 v[236:239], %[vl16], s[40:41] offset:2048 sc0 sc1\n\t" \
        "v_mfma_f32_16x16x32_bf16 v[44:47], v[168:171], v[200:203], v[44:47]\n\t" \
        "global_load_dwordx4 v[240:243], %[vl16], s[40:41] offset:3072 sc0 sc1\n\t" \
        "v_mfma_f32_16x16x32_bf16 v[12:15], v[168:171], v[196:199], v[12:15]\n\t" \
        "s_add_u32 s40, s40, 16384\n\t" \
        "s_addc_u32 s41, s41, 0\n\t" \
        "v_mfma_f32_16x16x32_bf16 v[12:15], v[164:167], v[180:183], v[12:15]\n\t" \
        "s_waitcnt lgkmcnt(2)\n\t" \
        "v_mfma_f32_16x16x32_bf16 v[16:19], v[172:175], v[180:183], v[16:19]\n\t" \
        "ds_read_b128 v[164:167], %[vaw] offset:25600\n\t" \
        "s_waitcnt lgkmcnt(2)\n\t" \
        "v_mfma_f32_16x16x32_bf16 v[16:19], v[176:179], v[196:199], v[16:19]\n\t" \
        "v_mfma_f32_16x16x32_bf16 v[48:51], v[176:179], v[200:203], v[48:51]\n\t" \
        "ds_read_b128 v[168:171], %[vaw] offset:16384\n\t" \
        "v_mfma_f32_16x16x32_bf16 v[48:51], v[172:175], v[184:187], v[48:51]\n\t" \
        "ds_write_b32 %[afree], %[vt]\n\t" \
        "v_mfma_f32_16x16x32_bf16 v[80:83], v[172:175], v[188:191], v[80:83]\n\t" \
        "v_mfma_f32_16x16x32_bf16 v[80:83], v[176:179], v[204:207], v[80:83]\n\t" \
        "v_mfma_f32_16x16x32_bf16 v[112:115], v[176:179], v[208:211], v[112:115]\n\t" \
        "v_mfma_f32_16x16x32_bf16 v[112:115], v[172:175], v[192:195], v[112:115]\n\t" \
        "\nLT1%=:\n\t" \
        "s_waitcnt vmcnt(4) lgkmcnt(3)\n\t" \
        "v_mfma_f32_16x16x32_bf16 v[20:23], v[160:163], v[196:199], v[20:23]\n\t" \
        "ds_read_b128 v[172:175], %[vaw] offset:26624\n\t" \
        "s_waitcnt lgkmcnt(3)\n\t" \
        "v_mfma_f32_16x16x32_bf16 v[20:23], v[164:167], v[212:215], v[20:23]\n\t" \
        "v_mfma_f32_16x16x32_bf16 v[52:55], v[164:167], v[216:219], v[52:55]\n\t" \
        "ds_read_b128 v[176:179], %[vaw] offset:17408\n\t" \
        "v_mfma_f32_16x16x32_bf16 v[52:55], v[160:163], v[200:203], v[52:55]\n\t" \
        "v_mfma_f32_16x16x32_bf16 v[84:87], v[160:163], v[204:207], v[84:87]\n\t" \
        "v_add_u32 %[vt], 1, %[vt]\n\t" \
        "v_mfma_f32_16x16x32_bf16 v[84:87], v[164:167], v[220:223], v[84:87]\n\t" \
        "v_mfma_f32_16x16x32_bf16 v[116:119], v[164:167], v[224:227], v[116:119]\n\t" \
        "v_mfma_f32_16x16x32_bf16 v[116:119], v[160:163], v[208:211], v[116:119]\n\t" \
        "s_waitcnt lgkmcnt(3)\n\t" \
        "v_mfma_f32_16x16x32_bf16 v[120:123], v[168:171], v[208:211], v[120:123]\n\t" \
        "ds_read_b128 v[160:163], %[vaw] offset:27648\n\t" \
        "s_waitcnt lgkmcnt(2)\n\t" \
        "v_mfma_f32_16x16x32_bf16 v[120:123], v[172:175], v[224:227], v[120:123]\n\t" \
        "v_mfma_f32_16x16x32_bf16 v[88:91], v[172:175], v[220:223], v[88:91]\n\t" \
        "ds_read_b128 v[164:167], %[vaw] offset:18432\n\t" \
        "v_mfma_f32_16x16x32_bf16 v[88:91], v[168:171], v[204:207], v[88:91]\n\t" \
        "v_mfma_f32_16x16x32_bf16 v[56:59], v[168:171], v[200:203], v[56:59]\n\t" \
        "ds_read_b32 %[vf0], %[ago]\n\t" \
        "v_mfma_f32_16x16x32_bf16 v[56:59], v[172:175], v[216:219], v[56:59]\n\t" \
        "v_mfma_f32_16x16x32_bf16 v[24:27], v[172:175], v[212:215], v[24:27]\n\t" \
        "v_mfma_f32_16x16x32_bf16 v[24:27], v[168:171], v[196:199], v[24:27]\n\t" \
        "s_waitcnt lgkmcnt(3)\n\t" \
        "v_mfma_f32_16x16x32_bf16 v[28:31], v[176:179], v[196:199], v[28:31]\n\t" \
        "ds_read_b128 v[168:171], %[vaw] offset:28672\n\t" \
        "s_waitcnt lgkmcnt(3)\n\t" \
        "v_mfma_f32_16x16x32_bf16 v[28:31], v[160:163], v[212:215], v[28:31]\n\t" \
        "v_mfma_f32_16x16x32_bf16 v[60:63], v[160:163], v[216:219], v[60:63]\n\t" \
        "ds_read_b128 v[172:175], %[vaw] offset:19456\n\t" \
        "v_mfma_f32_16x16x32_bf16 v[60:63], v[176:179], v[200:203], v[60:63]\n\t" \
        "v_mfma_f32_16x16x32_bf16 v[92:95], v[176:179], v[204:207], v[92:95]\n\t" \
        "s_waitcnt lgkmcnt(2)\n\t" \
        "v_mfma_f32_16x16x32_bf16 v[92:95], v[160:163], v[220:223], v[92:95]\n\t" \
        "v_cmp_ge_u32 vcc, %[vt], %[vf0]\n\t" \
        "v_mfma_f32_16x16x32_bf16 v[124:127], v[160:163], v[224:227], v[124:127]\n\t" \
        "v_mfma_f32_16x16x32_bf16 v[124:127], v[176:179], v[208:211], v[124:127]\n\t" \
        "s_cbranch_vccnz LSLT1%=\n\t" \
        "v_mfma_f32_16x16x32_bf16 v[152:155], v[164:167], v[208:211], v[152:155]\n\t" \
        "ds_read_b128 v[176:179], %[vaw] offset:29696\n\t" \
        "s_waitcnt lgkmcnt(2)\n\t" \
        "v_mfma_f32_16x16x32_bf16 v[152:155], v[168:171], v[224:227], v[152:155]\n\t" \
        "v_mfma_f32_16x16x32_bf16 v[144:147], v[168:171], v[220:223], v[144:147]\n\t" \
        "ds_read_b128 v[160:163], %[vaw] offset:20480\n\t" \
        "v_mfma_f32_16x16x32_bf16 v[144:147], v[164:167], v[204:207], v[144:147]\n\t" \
        "v_mfma_f32_16x16x32_bf16 v[136:139], v[164:167], v[200:203], v[136:139]\n\t" \
        "v_mfma_f32_16x16x32_bf16 v[136:139], v[168:171], v[216:219], v[136:139]\n\t" \
        "v_mfma_f32_16x16x32_bf16 v[128:131], v[168:171], v[212:215], v[128:131]\n\t" \
        "v_mfma_f32_16x16x32_bf16 v[128:131], v[164:167], v[196:199], v[128:131]\n\t" \
        "s_waitcnt lgkmcnt(2)\n\t" \
        "v_mfma_f32_16x16x32_bf16 v[132:135], v[172:175], v[196:199], v[132:135]\n\t" \
        "ds_read_b128 v[164:167], %[vaw] offset:30720\n\t" \
        "s_waitcnt lgkmcnt(2)\n\t" \
        "v_mfma_f32_16x16x32_bf16 v[132:135], v[176:179], v[212:215], v[132:135]\n\t" \
        "v_mfma_f32_16x16x32_bf16 v[140:143], v[176:179], v[216:219], v[140:143]\n\t" \
        "ds_read_b128 v[168:171], %[vaw] offset:21504\n\t" \
        "v_mfma_f32_16x16x32_bf16 v[140:143], v[172:175], v[200:203], v[140:143]\n\t" \
        "ds_write_b32 %[afree], %[vt]\n\t" \
        "v_mfma_f32_16x16x32_bf16 v[148:151], v[172:175], v[204:207], v[148:151]\n\t" \
        "v_mfma_f32_16x16x32_bf16 v[148:151], v[176:179], v[220:223], v[148:151]\n\t" \
        "v_mfma_f32_16x16x32_bf16 v[156:159], v[176:179], v[224:227], v[156:159]\n\t" \
        "v_mfma_f32_16x16x32_bf16 v[156:159], v[172:175], v[208:211], v[156:159]\n\t" \
        "\nLT2%=:\n\t" \
        "s_waitcnt vmcnt(0) lgkmcnt(3)\n\t" \
        "v_mfma_f32_16x16x32_bf16 v[0:3], v[160:163], v[212:215], v[0:3]\n\t" \
        "ds_read_b128 v[172:175], %[vaw] offset:31744\n\t" \
        "s_waitcnt lgkmcnt(3)\n\t" \
        "v_mfma_f32_16x16x32_bf16 v[0:3], v[164:167], v[228:231], v[0:3]\n\t" \
        "v_mfma_f32_16x16x32_bf16 v[32:35], v[164:167], v[232:235], v[32:35]\n\t" \
        "ds_read_b128 v[176:179], %[vaw] offset:22528\n\t" \
        "v_mfma_f32_16x16x32_bf16 v[32:35], v[160:163], v[216:219], v[32:35]\n\t" \
        "v_mfma_f32_16x16x32_bf16 v[64:67], v[160:163], v[220:223], v[64:67]\n\t" \
        "v_add_u32 %[vt], 1, %[vt]\n\t" \
        "v_mfma_f32_16x16x32_bf16 v[64:67], v[164:167], v[236:239], v[64:67]\n\t" \
        "v_mfma_f32_16x16x32_bf16 v[96:99], v[164:167], v[240:243], v[96:99]\n\t" \
        "v_mfma_f32_16x16x32_bf16 v[96:99], v[160:163], v[224:227], v[96:99]\n\t" \
        "s_waitcnt lgkmcnt(3)\n\t" \
        "v_mfma_f32_16x16x32_bf16 v[100:103], v[168:171], v[224:227], v[100:103]\n\t" \
        "ds_read_b128 v[160:163], %[vaw] offset:32768\n\t" \
        "s_waitcnt lgkmcnt(2)\n\t" \
        "v_mfma_f32_16x16x32_bf16 v[100:103], v[172:175], v[240:243], v[100:103]\n\t" \
        "v_mfma_f32_16x16x32_bf16 v[68:71], v[172:175], v[236:239], v[68:71]\n\t" \
        "ds_read_b128 v[164:167], %[vaw] offset:23552\n\t" \
        "v_mfma_f32_16x16x32_bf16 v[68:71], v[168:171], v[220:223], v[68:71]\n\t" \
        "v_mfma_f32_16x16x32_bf16 v[36:39], v[168:171], v[216:219], v[36:39]\n\t" \
        "v_mfma_f32_16x16x32_bf16 v[36:39], v[172:175], v[232:235], v[36:39]\n\t" \
        "v_mfma_f32_16x16x32_bf16 v[4:7], v[172:175], v[228:231], v[4:7]\n\t" \
        "v_mfma_f32_16x16x32_bf16 v[4:7], v[168:171], v[212:215], v[4:7]\n\t" \
        "s_waitcnt lgkmcnt(2)\n\t" \
        "v_mfma_f32_16x16x32_bf16 v[8:11], v[176:179], v[212:215], v[8:11]\n\t" \
        "ds_read_b128 v[168:171], %[vaw] offset:33792\n\t" \
        "s_waitcnt lgkmcnt(2)\n\t" \
        "v_mfma_f32_16x16x32_bf16 v[8:11], v[160:163], v[228:231], v[8:11]\n\t" \
        "v_mfma_f32_16x16x32_bf16 v[40:43], v[160:163], v[232:235], v[40:43]\n\t" \
        "ds_read_b128 v[172:175], %[vaw] offset:24576\n\t" \
        "v_mfma_f32_16x16x32_bf16 v[40:43], v[176:179], v[216:219], v[40:43]\n\t" \
        "v_mfma_f32_16x16x32_bf16 v[72:75], v[176:179], v[220:223], v[72:75]\n\t" \
        "v_mfma_f32_16x16x32_bf16 v[72:75], v[160:163], v[236:239], v[72:75]\n\t" \
        "v_mfma_f32_16x16x32_bf16 v[104:107], v[160:163], v[240:243], v[104:107]\n\t" \
        "v_mfma_f32_16x16x32_bf16 v[104:107], v[176:179], v[224:227], v[104:107]\n\t" \
        "s_waitcnt lgkmcnt(2)\n\t" \
        "v_mfma_f32_16x16x32_bf16 v[108:111], v[164:167], v[224:227], v[108:111]\n\t" \
        "ds_read_b128 v[176:179], %[vaw] offset:34816\n\t" \
        "s_waitcnt lgkmcnt(2)\n\t" \
        "v_mfma_f32_16x16x32_bf16 v[108:111], v[168:171], v[240:243], v[108:111]\n\t" \
        "v_mfma_f32_16x16x32_bf16 v[76:79], v[168:171], v[236:239], v[76:79]\n\t" \
        "ds_read_b128 v[160:163], %[vaw] offset:35840\n\t" \
        "v_mfma_f32_16x16x32_bf16 v[76:79], v[164:167], v[220:223], v[76:79]\n\t" \
        "v_mfma_f32_16x16x32_bf16 v[44:47], v[164:167], v[216:219], v[44:47]\n\t" \
        "v_mfma_f32_16x16x32_bf16 v[44:47], v[168:171], v[232:235], v[44:47]\n\t" \
        "v_mfma_f32_16x16x32_bf16 v[12:15], v[168:171], v[228:231], v[12:15]\n\t" \
        "v_mfma_f32_16x16x32_bf16 v[12:15], v[164:167], v[212:215], v[12:15]\n\t" \
        "s_waitcnt lgkmcnt(2)\n\t" \
        "v_mfma_f32_16x16x32_bf16 v[16:19], v[172:175], v[212:215], v[16:19]\n\t" \
        "s_waitcnt lgkmcnt(2)\n\t" \
        "v_mfma_f32_16x16x32_bf16 v[16:19], v[176:179], v[228:231], v[16:19]\n\t" \
        "v_mfma_f32_16x16x32_bf16 v[48:51], v[176:179], v[232:235], v[48:51]\n\t" \
        "ds_read_b128 v[168:171], %[vaw] offset:36864\n\t" \
        "v_mfma_f32_16x16x32_bf16 v[48:51], v[172:175], v[216:219], v[48:51]\n\t" \
        "ds_write_b32 %[afree], %[vt]\n\t" \
        "v_mfma_f32_16x16x32_bf16 v[80:83], v[172:175], v[220:223], v[80:83]\n\t" \
        "v_mfma_f32_16x16x32_bf16 v[80:83], v[176:179], v[236:239], v[80:83]\n\t" \
        "v_mfma_f32_16x16x32_bf16 v[112:115], v[176:179], v[240:243], v[112:115]\n\t" \
        "v_mfma_f32_16x16x32_bf16 v[112:115], v[172:175], v[224:227], v[112:115]\n\t" \
        "v_mov_b32 %[vf0], 0\n\t" \
        "s_waitcnt lgkmcnt(2)\n\t" \
        "v_mfma_f32_16x16x32_bf16 v[20:23], v[160:163], v[228:231], v[20:23]\n\t" \
        "ds_read_b128 v[176:179], %[vaw] offset:37888\n\t" \
        "v_mfma_f32_16x16x32_bf16 v[52:55], v[160:163], v[232:235], v[52:55]\n\t" \
        "ds_read_b128 v[164:167], %[vaw] offset:38912\n\t" \
        "v_mfma_f32_16x16x32_bf16 v[84:87], v[160:163], v[236:239], v[84:87]\n\t" \
        "ds_read_b128 v[172:175], %[vaw] offset:39936\n\t" \
        "v_mfma_f32_16x16x32_bf16 v[116:119], v[160:163], v[240:243], v[116:119]\n\t" \
        "v_mov_b32 %[vt], 0\n\t" \
        "s_waitcnt lgkmcnt(3)\n\t" \
        "v_mfma_f32_16x16x32_bf16 v[120:123], v[168:171], v[240:243], v[120:123]\n\t" \
        "v_max3_f32 %[vf0], %[vf0], v0, v1\n\t" \
        "v_max3_f32 %[vt], %[vt], v2, v3\n\t" \
        "v_mfma_f32_16x16x32_bf16 v[88:91], v[168:171], v[236:239], v[88:91]\n\t" \
        "v_max3_f32 %[vf0], %[vf0], v32, v33\n\t" \
        "v_max3_f32 %[vt], %[vt], v34, v35\n\t" \
        "v_mfma_f32_16x16x32_bf16 v[56:59], v[168:171], v[232:235], v[56:59]\n\t" \
        "v_max3_f32 %[vf0], %[vf0], v64, v65\n\t" \
        "v_max3_f32 %[vt], %[vt], v66, v67\n\t" \
        "v_mfma_f32_16x16x32_bf16 v[24:27], v[168:171], v[228:231], v[24:27]\n\t" \
        "s_waitcnt lgkmcnt(2)\n\t" \
        "v_mfma_f32_16x16x32_bf16 v[28:31], v[176:179], v[228:231], v[28:31]\n\t" \
        "v_max3_f32 %[vf0], %[vf0], v96, v97\n\t" \
        "v_max3_f32 %[vt], %[vt], v98, v99\n\t" \
        "v_mfma_f32_16x16x32_bf16 v[60:63], v[176:179], v[232:235], v[60:63]\n\t" \
        "v_max3_f32 %[vf0], %[vf0], v4, v5\n\t" \
        "v_max3_f32 %[vt], %[vt], v6, v7\n\t" \
        "v_mfma_f32_16x16x32_bf16 v[92:95], v[176:179], v[236:239], v[92:95]\n\t" \
        "v_max3_f32 %[vf0], %[vf0], v36, v37\n\t" \
        "v_max3_f32 %[vt], %[vt], v38, v39\n\t" \
        "v_mfma_f32_16x16x32_bf16 v[124:127], v[176:179], v[240:243], v[124:127]\n\t" \
        "s_waitcnt lgkmcnt(1)\n\t" \
        "v_mfma_f32_16x16x32_bf16 v[152:155], v[164:167], v[240:243], v[152:155]\n\t" \
        "v_max3_f32 %[vf0], %[vf0], v68, v69\n\t" \
        "v_max3_f32 %[vt], %[vt], v70, v71\n\t" \
        "v_mfma_f32_16x16x32_bf16 v[144:147], v[164:167], v[236:239], v[144:147]\n\t" \
        "v_max3_f32 %[vf0], %[vf0], v100, v101\n\t" \
        "v_max3_f32 %[vt], %[vt], v102, v103\n\t" \
        "v_mfma_f32_16x16x32_bf16 v[136:139], v[164:167], v[232:235], v[136:139]\n\t" \
        "v_max3_f32 %[vf0], %[vf0], v8, v9\n\t" \
        "v_max3_f32 %[vt], %[vt], v10, v11\n\t" \
        "v_mfma_f32_16x16x32_bf16 v[128:131], v[164:167], v[228:231], v[128:131]\n\t" \
        "s_waitcnt lgkmcnt(0)\n\t" \
        "v_mfma_f32_16x16x32_bf16 v[132:135], v[172:175], v[228:231], v[132:135]\n\t" \
        "v_max3_f32 %[vf0], %[vf0], v40, v41\n\t" \
        "v_max3_f32 %[vt], %[vt], v42, v43\n\t" \
        "v_mfma_f32_16x16x32_bf16 v[140:143], v[172:175], v[232:235], v[140:143]\n\t" \
        "v_max3_f32 %[vf0], %[vf0], v72, v73\n\t" \
        "v_max3_f32 %[vt], %[vt], v74, v75\n\t" \
        "v_mfma_f32_16x16x32_bf16 v[148:151], v[172:175], v[236:239], v[148:151]\n\t" \
        "v_max3_f32 %[vf0], %[vf0], v104, v105\n\t" \
        "v_max3_f32 %[vt], %[vt], v106, v107\n\t" \
        "v_mfma_f32_16x16x32_bf16 v[156:159], v[172:175], v[240:243], v[156:159]\n\t" \
        "v_max3_f32 %[vf0], %[vf0], v12, v13\n\t" \
        "v_max3_f32 %[vt], %[vt], v14, v15\n\t" \
        "v_max3_f32 %[vf0], %[vf0], v44, v45\n\t" \
        "v_max3_f32 %[vt], %[vt], v46, v47\n\t" \
        "v_max3_f32 %[vf0], %[vf0], v76, v77\n\t" \
        "v_max3_f32 %[vt], %[vt], v78, v79\n\t" \
        "v_max3_f32 %[vf0], %[vf0], v108, v109\n\t" \
        "v_max3_f32 %[vt], %[vt], v110, v111\n\t" \
        "v_max3_f32 %[vf0], %[vf0], v16, v17\n\t" \
        "v_max3_f32 %[vt], %[vt], v18, v19\n\t" \
        "v_max3_f32 %[vf0], %[vf0], v48, v49\n\t" \
        "v_max3_f32 %[vt], %[vt], v50, v51\n\t" \
        "v_max3_f32 %[vf0], %[vf0], v80, v81\n\t" \
        "v_max3_f32 %[vt], %[vt], v82, v83\n\t" \
        "v_max3_f32 %[vf0], %[vf0], v112, v113\n\t" \
        "v_max3_f32 %[vt], %[vt], v114, v115\n\t" \
        "v_max3_f32 %[vf0], %[vf0], v20, v21\n\t" \
        "v_max3_f32 %[vt], %[vt], v22, v23\n\t" \
        "v_max3_f32 %[vf0], %[vf0], v52, v53\n\t" \
        "v_max3_f32 %[vt], %[vt], v54, v55\n\t" \
        "v_max3_f32 %[vf0], %[vf0], v84, v85\n\t" \
        "v_max3_f32 %[vt], %[vt], v86, v87\n\t" \
        "v_max3_f32 %[vf0], %[vf0], v116, v117\n\t" \
        "v_max3_f32 %[vt], %[vt], v118, v119\n\t" \
        "v_max3_f32 %[vf0], %[vf0], v24, v25\n\t" \
        "v_max3_f32 %[vt], %[vt], v26, v27\n\t" \
        "v_max3_f32 %[vf0], %[vf0], v56, v57\n\t" \
        "v_max3_f32 %[vt], %[vt], v58, v59\n\t" \
        "v_max3_f32 %[vf0], %[vf0], v88, v89\n\t" \
        "v_max3_f32 %[vt], %[vt], v90, v91\n\t" \
        "v_max3_f32 %[vf0], %[vf0], v120, v121\n\t" \
        "v_max3_f32 %[vt], %[vt], v122, v123\n\t" \
        "v_max3_f32 %[vf0], %[vf0], v28, v29\n\t" \
        "v_max3_f32 %[vt], %[vt], v30, v31\n\t" \
        "v_max3_f32 %[vf0], %[vf0], v60, v61\n\t" \
        "v_max3_f32 %[vt], %[vt], v62, v63\n\t" \
        "v_max3_f32 %[vf0], %[vf0], v92, v93\n\t" \
        "v_max3_f32 %[vt], %[vt], v94, v95\n\t" \
        "v_max3_f32 %[vf0], %[vf0], v124, v125\n\t" \
        "v_max3_f32 %[vt], %[vt], v126, v127\n\t" \
        "v_max3_f32 %[vf0], %[vf0], v128, v129\n\t" \
        "v_max3_f32 %[vt], %[vt], v130, v131\n\t" \
        "v_max3_f32 %[vf0], %[vf0], v136, v137\n\t" \
        "v_max3_f32 %[vt], %[vt], v138, v139\n\t" \
        "v_max3_f32 %[vf0], %[vf0], v144, v145\n\t" \
        "v_max3_f32 %[vt], %[vt], v146, v147\n\t" \
        "v_max3_f32 %[vf0], %[vf0], v152, v153\n\t" \
        "v_max3_f32 %[vt], %[vt], v154, v155\n\t" \
        "v_max3_f32 %[vf0], %[vf0], v132, v133\n\t" \
        "v_max3_f32 %[vt], %[vt], v134, v135\n\t" \
        "v_max3_f32 %[vf0], %[vf0], v140, v141\n\t" \
        "v_max3_f32 %[vt], %[vt], v142, v143\n\t" \
        "v_max3_f32 %[vf0], %[vf0], v148, v149\n\t" \
        "v_max3_f32 %[vt], %[vt], v150, v151\n\t" \
        "v_max3_f32 %[vf0], %[vf0], v156, v157\n\t" \
        "v_max3_f32 %[vt], %[vt], v158, v159\n\t" \
        "v_max_f32 %[vf0], %[vf0], %[vt]\n\t" \
        "s_branch LDONE%=\n\t" \
        "\nLSLM0%=:\n\t" \
        "s_waitcnt lgkmcnt(0)\n\t" \
        "v_mfma_f32_16x16x32_bf16 v[108:111], v[164:167], v[192:195], v[108:111]\n\t" \
        "ds_read_b128 v[176:179], %[vaw] offset:14336\n\t" \
        "v_mfma_f32_16x16x32_bf16 v[108:111], v[168:171], v[208:211], v[108:111]\n\t" \
        "v_mfma_f32_16x16x32_bf16 v[76:79], v[168:171], v[204:207], v[76:79]\n\t" \
        "v_mfma_f32_16x16x32_bf16 v[76:79], v[164:167], v[188:191], v[76:79]\n\t" \
        "v_mfma_f32_16x16x32_bf16 v[44:47], v[164:167], v[184:187], v[44:47]\n\t" \
        "v_mfma_f32_16x16x32_bf16 v[44:47], v[168:171], v[200:203], v[44:47]\n\t" \
        "v_mfma_f32_16x16x32_bf16 v[12:15], v[168:171], v[196:199], v[12:15]\n\t" \
        "v_mfma_f32_16x16x32_bf16 v[12:15], v[164:167], v[180:183], v[12:15]\n\t" \
        "s_waitcnt lgkmcnt(0)\n\t" \
        "ds_write_b32 %[afree], %[vt]\n\t" \
        "v_mfma_f32_16x16x32_bf16 v[16:19], v[172:175], v[180:183], v[16:19]\n\t" \
        "v_mfma_f32_16x16x32_bf16 v[16:19], v[176:179], v[196:199], v[16:19]\n\t" \
        "v_mfma_f32_16x16x32_bf16 v[48:51], v[176:179], v[200:203], v[48:51]\n\t" \
        "v_mfma_f32_16x16x32_bf16 v[48:51], v[172:175], v[184:187], v[48:51]\n\t" \
        "v_mfma_f32_16x16x32_bf16 v[80:83], v[172:175], v[188:191], v[80:83]\n\t" \
        "v_mfma_f32_16x16x32_bf16 v[80:83], v[176:179], v[204:207], v[80:83]\n\t" \
        "v_mfma_f32_16x16x32_bf16 v[112:115], v[176:179], v[208:211], v[112:115]\n\t" \
        "v_mfma_f32_16x16x32_bf16 v[112:115], v[172:175], v[192:195], v[112:115]\n\t" \
        "\nLSPM0%=:\n\t" \
        "s_sleep 1\n\t" \
        "ds_read_b32 %[vf0], %[ago]\n\t" \
        "s_waitcnt lgkmcnt(0)\n\t" \
        "v_cmp_ge_u32 vcc, %[vt], %[vf0]\n\t" \
        "s_cbranch_vccnz LSPM0%=\n\t" \
        "global_load_dwordx4 v[228:231], %[vl16], s[40:41] offset:0 sc0 sc1\n\t" \
        "global_load_dwordx4 v[232:235], %[vl16], s[40:41] offset:1024 sc0 sc1\n\t" \
        "global_load_dwordx4 v[236:239], %[vl16], s[40:41] offset:2048 sc0 sc1\n\t" \
        "global_load_dwordx4 v[240:243], %[vl16], s[40:41] offset:3072 sc0 sc1\n\t" \
        "s_add_u32 s40, s40, 16384\n\t" \
        "s_addc_u32 s41, s41, 0\n\t" \
        "ds_read_b128 v[160:163], %[vaw] offset:15360\n\t" \
        "ds_read_b128 v[164:167], %[vaw] offset:25600\n\t" \
        "ds_read_b128 v[168:171], %[vaw] offset:16384\n\t" \
        "s_waitcnt lgkmcnt(0)\n\t" \
        "s_branch LM1%=\n\t" \
        "\nLSLM1%=:\n\t" \
        "s_waitcnt lgkmcnt(0)\n\t" \
        "v_mfma_f32_16x16x32_bf16 v[152:155], v[164:167], v[208:211], v[152:155]\n\t" \
        "ds_read_b128 v[176:179], %[vaw] offset:29696\n\t" \
        "v_mfma_f32_16x16x32_bf16 v[152:155], v[168:171], v[224:227], v[152:155]\n\t" \
        "v_mfma_f32_16x16x32_bf16 v[144:147], v[168:171], v[220:223], v[144:147]\n\t" \
        "v_mfma_f32_16x16x32_bf16 v[144:147], v[164:167], v[204:207], v[144:147]\n\t" \
        "v_mfma_f32_16x16x32_bf16 v[136:139], v[164:167], v[200:203], v[136:139]\n\t" \
        "v_mfma_f32_16x16x32_bf16 v[136:139], v[168:171], v[216:219], v[136:139]\n\t" \
        "v_mfma_f32_16x16x32_bf16 v[128:131], v[168:171], v[212:215], v[128:131]\n\t" \
        "v_mfma_f32_16x16x32_bf16 v[128:131], v[164:167], v[196:199], v[128:131]\n\t" \
        "s_waitcnt lgkmcnt(0)\n\t" \
        "ds_write_b32 %[afree], %[vt]\n\t" \
        "v_mfma_f32_16x16x32_bf16 v[132:135], v[172:175], v[196:199], v[132:135]\n\t" \
        "v_mfma_f32_16x16x32_bf16 v[132:135], v[176:179], v[212:215], v[132:135]\n\t" \
        "v_mfma_f32_16x16x32_bf16 v[140:143], v[176:179], v[216:219], v[140:143]\n\t" \
        "v_mfma_f32_16x16x32_bf16 v[140:143], v[172:175], v[200:203], v[140:143]\n\t" \
        "v_mfma_f32_16x16x32_bf16 v[148:151], v[172:175], v[204:207], v[148:151]\n\t" \
        "v_mfma_f32_16x16x32_bf16 v[148:151], v[176:179], v[220:223], v[148:151]\n\t" \
        "v_mfma_f32_16x16x32_bf16 v[156:159], v[176:179], v[224:227], v[156:159]\n\t" \
        "v_mfma_f32_16x16x32_bf16 v[156:159], v[172:175], v[208:211], v[156:159]\n\t" \
        "\nLSPM1%=:\n\t" \
        "s_sleep 1\n\t" \
        "ds_read_b32 %[vf0], %[ago]\n\t" \
        "s_waitcnt lgkmcnt(0)\n\t" \
        "v_cmp_ge_u32 vcc, %[vt], %[vf0]\n\t" \
        "s_cbranch_vccnz LSPM1%=\n\t" \
        "global_load_dwordx4 v[180:183], %[vl16], s[40:41] offset:0 sc0 sc1\n\t" \
        "global_load_dwordx4 v[184:187], %[vl16], s[40:41] offset:1024 sc0 sc1\n\t" \
        "global_load_dwordx4 v[188:191], %[vl16], s[40:41] offset:2048 sc0 sc1\n\t" \
        "global_load_dwordx4 v[192:195], %[vl16], s[40:41] offset:3072 sc0 sc1\n\t" \
        "s_add_u32 s40, s40, 16384\n\t" \
        "s_addc_u32 s41, s41, 0\n\t" \
        "ds_read_b128 v[160:163], %[vaw] offset:20480\n\t" \
        "ds_read_b128 v[164:167], %[vaw] offset:30720\n\t" \
        "ds_read_b128 v[168:171], %[vaw] offset:21504\n\t" \
        "s_waitcnt lgkmcnt(0)\n\t" \
        "s_branch LM2%=\n\t" \
        "\nLSLM2%=:\n\t" \
        "s_waitcnt lgkmcnt(0)\n\t" \
        "v_mfma_f32_16x16x32_bf16 v[108:111], v[164:167], v[224:227], v[108:111]\n\t" \
        "ds_read_b128 v[176:179], %[vaw] offset:34816\n\t" \
        "v_mfma_f32_16x16x32_bf16 v[108:111], v[168:171], v[240:243], v[108:111]\n\t" \
        "v_mfma_f32_16x16x32_bf16 v[76:79], v[168:171], v[236:239], v[76:79]\n\t" \
        "v_mfma_f32_16x16x32_bf16 v[76:79], v[164:167], v[220:223], v[76:79]\n\t" \
        "v_mfma_f32_16x16x32_bf16 v[44:47], v[164:167], v[216:219], v[44:47]\n\t" \
        "v_mfma_f32_16x16x32_bf16 v[44:47], v[168:171], v[232:235], v[44:47]\n\t" \
        "v_mfma_f32_16x16x32_bf16 v[12:15], v[168:171], v[228:231], v[12:15]\n\t" \
        "v_mfma_f32_16x16x32_bf16 v[12:15], v[164:167], v[212:215], v[12:15]\n\t" \
        "s_waitcnt lgkmcnt(0)\n\t" \
        "ds_write_b32 %[afree], %[vt]\n\t" \
        "v_mfma_f32_16x16x32_bf16 v[16:19], v[172:175], v[212:215], v[16:19]\n\t" \
        "v_mfma_f32_16x16x32_bf16 v[16:19], v[176:179], v[228:231], v[16:19]\n\t" \
        "v_mfma_f32_16x16x32_bf16 v[48:51], v[176:179], v[232:235], v[48:51]\n\t" \
        "v_mfma_f32_16x16x32_bf16 v[48:51], v[172:175], v[216:219], v[48:51]\n\t" \
        "v_mfma_f32_16x16x32_bf16 v[80:83], v[172:175], v[220:223], v[80:83]\n\t" \
        "v_mfma_f32_16x16x32_bf16 v[80:83], v[176:179], v[236:239], v[80:83]\n\t" \
        "v_mfma_f32_16x16x32_bf16 v[112:115], v[176:179], v[240:243], v[112:115]\n\t" \
        "v_mfma_f32_16x16x32_bf16 v[112:115], v[172:175], v[224:227], v[112:115]\n\t" \
        "\nLSPM2%=:\n\t" \
        "s_sleep 1\n\t" \
        "ds_read_b32 %[vf0], %[ago]\n\t" \
        "s_waitcnt lgkmcnt(0)\n\t" \
        "v_cmp_ge_u32 vcc, %[vt], %[vf0]\n\t" \
        "s_cbranch_vccnz LSPM2%=\n\t" \
        "global_load_dwordx4 v[196:199], %[vl16], s[40:41] offset:0 sc0 sc1\n\t" \
        "global_load_dwordx4 v[200:203], %[vl16], s[40:41] offset:1024 sc0 sc1\n\t" \
        "global_load_dwordx4 v[204:207], %[vl16], s[40:41] offset:2048 sc0 sc1\n\t" \
        "global_load_dwordx4 v[208:211], %[vl16], s[40:41] offset:3072 sc0 sc1\n\t" \
        "s_add_u32 s40, s40, 16384\n\t" \
        "s_addc_u32 s41, s41, 0\n\t" \
        "ds_read_b128 v[160:163], %[vaw] offset:35840\n\t" \
        "ds_read_b128 v[164:167], %[vax] offset:5120\n\t" \
        "ds_read_b128 v[168:171], %[vaw] offset:36864\n\t" \
        "s_waitcnt lgkmcnt(0)\n\t" \
        "s_branch LM3%=\n\t" \
        "\nLSLM3%=:\n\t" \
        "s_waitcnt lgkmcnt(0)\n\t" \
        "v_mfma_f32_16x16x32_bf16 v[152:155], v[164:167], v[240:243], v[152:155]\n\t" \
        "ds_read_b128 v[176:179], %[vax] offset:9216\n\t" \
        "v_mfma_f32_16x16x32_bf16 v[152:155], v[168:171], v[192:195], v[152:155]\n\t" \
        "v_mfma_f32_16x16x32_bf16 v[144:147], v[168:171], v[188:191], v[144:147]\n\t" \
        "v_mfma_f32_16x16x32_bf16 v[144:147], v[164:167], v[236:239], v[144:147]\n\t" \
        "v_mfma_f32_16x16x32_bf16 v[136:139], v[164:167], v[232:235], v[136:139]\n\t" \
        "v_mfma_f32_16x16x32_bf16 v[136:139], v[168:171], v[184:187], v[136:139]\n\t" \
        "v_mfma_f32_16x16x32_bf16 v[128:131], v[168:171], v[180:183], v[128:131]\n\t" \
        "v_mfma_f32_16x16x32_bf16 v[128:131], v[164:167], v[228:231], v[128:131]\n\t" \
        "s_waitcnt lgkmcnt(0)\n\t" \
        "ds_write_b32 %[afree], %[vt]\n\t" \
        "v_mfma_f32_16x16x32_bf16 v[132:135], v[172:175], v[228:231], v[132:135]\n\t" \
        "v_mfma_f32_16x16x32_bf16 v[132:135], v[176:179], v[180:183], v[132:135]\n\t" \
        "v_mfma_f32_16x16x32_bf16 v[140:143], v[176:179], v[184:187], v[140:143]\n\t" \
        "v_mfma_f32_16x16x32_bf16 v[140:143], v[172:175], v[232:235], v[140:143]\n\t" \
        "v_mfma_f32_16x16x32_bf16 v[148:151], v[172:175], v[236:239], v[148:151]\n\t" \
        "v_mfma_f32_16x16x32_bf16 v[148:151], v[176:179], v[188:191], v[148:151]\n\t" \
        "v_mfma_f32_16x16x32_bf16 v[156:159], v[176:179], v[192:195], v[156:159]\n\t" \
        "v_mfma_f32_16x16x32_bf16 v[156:159], v[172:175], v[240:243], v[156:159]\n\t" \
        "\nLSPM3%=:\n\t" \
        "s_sleep 1\n\t" \
        "ds_read_b32 %[vf0], %[ago]\n\t" \
        "s_waitcnt lgkmcnt(0)\n\t" \
        "v_cmp_ge_u32 vcc, %[vt], %[vf0]\n\t" \
        "s_cbranch_vccnz LSPM3%=\n\t" \
        "global_load_dwordx4 v[212:215], %[vl16], s[40:41] offset:0 sc0 sc1\n\t" \
        "global_load_dwordx4 v[216:219], %[vl16], s[40:41] offset:1024 sc0 sc1\n\t" \
        "global_load_dwordx4 v[220:223], %[vl16], s[40:41] offset:2048 sc0 sc1\n\t" \
        "global_load_dwordx4 v[224:227], %[vl16], s[40:41] offset:3072 sc0 sc1\n\t" \
        "s_add_u32 s40, s40, 16384\n\t" \
        "s_addc_u32 s41, s41, 0\n\t" \
        "ds_read_b128 v[160:163], %[vax] offset:0\n\t" \
        "ds_read_b128 v[164:167], %[vax] offset:10240\n\t" \
        "ds_read_b128 v[168:171], %[vax] offset:1024\n\t" \
        "v_mov_b32 %[vaw], %[vax]\n\t" \
        "s_waitcnt lgkmcnt(0)\n\t" \
        "s_sub_u32 %[ss], %[ss], 1\n\t" \
        "s_cmp_lg_u32 %[ss], 0\n\t" \
        "s_cbranch_scc1 LM0%=\n\t" \
        "s_branch LT0%=\n\t" \
        "\nLSLT0%=:\n\t" \
        "s_waitcnt lgkmcnt(0)\n\t" \
        "v_mfma_f32_16x16x32_bf16 v[108:111], v[164:167], v[192:195], v[108:111]\n\t" \
        "ds_read_b128 v[176:179], %[vaw] offset:14336\n\t" \
        "v_mfma_f32_16x16x32_bf16 v[108:111], v[168:171], v[208:211], v[108:111]\n\t" \
        "v_mfma_f32_16x16x32_bf16 v[76:79], v[168:171], v[204:207], v[76:79]\n\t" \
        "v_mfma_f32_16x16x32_bf16 v[76:79], v[164:167], v[188:191], v[76:79]\n\t" \
        "v_mfma_f32_16x16x32_bf16 v[44:47], v[164:167], v[184:187], v[44:47]\n\t" \
        "v_mfma_f32_16x16x32_bf16 v[44:47], v[168:171], v[200:203], v[44:47]\n\t" \
        "v_mfma_f32_16x16x32_bf16 v[12:15], v[168:171], v[196:199], v[12:15]\n\t" \
        "v_mfma_f32_16x16x32_bf16 v[12:15], v[164:167], v[180:183], v[12:15]\n\t" \
        "s_waitcnt lgkmcnt(0)\n\t" \
        "ds_write_b32 %[afree], %[vt]\n\t" \
        "v_mfma_f32_16x16x32_bf16 v[16:19], v[172:175], v[180:183], v[16:19]\n\t" \
        "v_mfma_f32_16x16x32_bf16 v[16:19], v[176:179], v[196:199], v[16:19]\n\t" \
        "v_mfma_f32_16x16x32_bf16 v[48:51], v[176:179], v[200:203], v[48:51]\n\t" \
        "v_mfma_f32_16x16x32_bf16 v[48:51], v[172:175], v[184:187], v[48:51]\n\t" \
        "v_mfma_f32_16x16x32_bf16 v[80:83], v[172:175], v[188:191], v[80:83]\n\t" \
        "v_mfma_f32_16x16x32_bf16 v[80:83], v[176:179], v[204:207], v[80:83]\n\t" \
        "v_mfma_f32_16x16x32_bf16 v[112:115], v[176:179], v[208:211], v[112:115]\n\t" \
        "v_mfma_f32_16x16x32_bf16 v[112:115], v[172:175], v[192:195], v[112:115]\n\t" \
        "\nLSPT0%=:\n\t" \
        "s_sleep 1\n\t" \
        "ds_read_b32 %[vf0], %[ago]\n\t" \
        "s_waitcnt lgkmcnt(0)\n\t" \
        "v_cmp_ge_u32 vcc, %[vt], %[vf0]\n\t" \
        "s_cbranch_vccnz LSPT0%=\n\t" \
        "global_load_dwordx4 v[228:231], %[vl16], s[40:41] offset:0 sc0 sc1\n\t" \
        "global_load_dwordx4 v[232:235], %[vl16], s[40:41] offset:1024 sc0 sc1\n\t" \
        "global_load_dwordx4 v[236:239], %[vl16], s[40:41] offset:2048 sc0 sc1\n\t" \
        "global_load_dwordx4 v[240:243], %[vl16], s[40:41] offset:3072 sc0 sc1\n\t" \
        "s_add_u32 s40, s40, 16384\n\t" \
        "s_addc_u32 s41, s41, 0\n\t" \
        "ds_read_b128 v[160:163], %[vaw] offset:15360\n\t" \
        "ds_read_b128 v[164:167], %[vaw] offset:25600\n\t" \
        "ds_read_b128 v[168:171], %[vaw] offset:16384\n\t" \
        "s_waitcnt lgkmcnt(0)\n\t" \
        "s_branch LT1%=\n\t" \
        "\nLSLT1%=:\n\t" \
        "s_waitcnt lgkmcnt(0)\n\t" \
        "v_mfma_f32_16x16x32_bf16 v[152:155], v[164:167], v[208:211], v[152:155]\n\t" \
        "ds_read_b128 v[176:179], %[vaw] offset:29696\n\t" \
        "v_mfma_f32_16x16x32_bf16 v[152:155], v[168:171], v[224:227], v[152:155]\n\t" \
        "v_mfma_f32_16x16x32_bf16 v[144:147], v[168:171], v[220:223], v[144:147]\n\t" \
        "v_mfma_f32_16x16x32_bf16 v[144:147], v[164:167], v[204:207], v[144:147]\n\t" \
        "v_mfma_f32_16x16x32_bf16 v[136:139], v[164:167], v[200:203], v[136:139]\n\t" \
        "v_mfma_f32_16x16x32_bf16 v[136:139], v[168:171], v[216:219], v[136:139]\n\t" \
        "v_mfma_f32_16x16x32_bf16 v[128:131], v[168:171], v[212:215], v[128:131]\n\t" \
        "v_mfma_f32_16x16x32_bf16 v[128:131], v[164:167], v[196:199], v[128:131]\n\t" \
        "s_waitcnt lgkmcnt(0)\n\t" \
        "ds_write_b32 %[afree], %[vt]\n\t" \
        "v_mfma_f32_16x16x32_bf16 v[132:135], v[172:175], v[196:199], v[132:135]\n\t" \
        "v_mfma_f32_16x16x32_bf16 v[132:135], v[176:179], v[212:215], v[132:135]\n\t" \
        "v_mfma_f32_16x16x32_bf16 v[140:143], v[176:179], v[216:219], v[140:143]\n\t" \
        "v_mfma_f32_16x16x32_bf16 v[140:143], v[172:175], v[200:203], v[140:143]\n\t" \
        "v_mfma_f32_16x16x32_bf16 v[148:151], v[172:175], v[204:207], v[148:151]\n\t" \
        "v_mfma_f32_16x16x32_bf16 v[148:151], v[176:179], v[220:223], v[148:151]\n\t" \
        "v_mfma_f32_16x16x32_bf16 v[156:159], v[176:179], v[224:227], v[156:159]\n\t" \
        "v_mfma_f32_16x16x32_bf16 v[156:159], v[172:175], v[208:211], v[156:159]\n\t" \
        "\nLSPT1%=:\n\t" \
        "s_sleep 1\n\t" \
        "ds_read_b32 %[vf0], %[ago]\n\t" \
        "s_waitcnt lgkmcnt(0)\n\t" \
        "v_cmp_ge_u32 vcc, %[vt], %[vf0]\n\t" \
        "s_cbranch_vccnz LSPT1%=\n\t" \
        "ds_read_b128 v[160:163], %[vaw] offset:20480\n\t" \
        "ds_read_b128 v[164:167], %[vaw] offset:30720\n\t" \
        "ds_read_b128 v[168:171], %[vaw] offset:21504\n\t" \
        "s_waitcnt lgkmcnt(0)\n\t" \
        "s_branch LT2%=\n\t" \
        "\nLDONE%=:\n\t" \
        "s_nop 1\n\t"
#define PULL \
        "s_mov_b64 s[64:65], %[pk]\n\t" \
        "s_mov_b64 s[66:67], %[ov]\n\t" \
        "s_mov_b64 s[68:69], %[hp]\n\t" \
        "s_mov_b64 s[70:71], %[xb]\n\t" \
        "s_mov_b64 s[72:73], %[fx]\n\t" \
        "s_mov_b64 s[52:53], %[dum]\n\t" \
        "s_mov_b32 s50, %[meta]\n\t" \
        "v_lshlrev_b32 v8, 4, %[lane]\n\t" \
        "v_mul_u32_u24 v10, 12, %[lane]\n\t" \
        "v_lshlrev_b32 v11, 2, %[lane]\n\t" \
        "v_add_u32 v9, 0x1000, v8\n\t" \
        "v_add_u32 v10, 0x1000, v10\n\t" \
        "v_add_u32 v11, 0x1600, v11\n\t" \
        "s_mul_i32 s62, %[cur], 5888\n\t" \
        "s_add_u32 s44, s64, s62\n\t" \
        "s_addc_u32 s45, s65, 0\n\t" \
        "global_load_dwordx4 v[20:23], v8, s[44:45] offset:0 sc0 sc1\n\t" \
        "global_load_dwordx4 v[24:27], v8, s[44:45] offset:1024 sc0 sc1\n\t" \
        "global_load_dwordx4 v[28:31], v8, s[44:45] offset:2048 sc0 sc1\n\t" \
        "global_load_dwordx4 v[32:35], v8, s[44:45] offset:3072 sc0 sc1\n\t" \
        "s_bitcmp1_b32 s50, 23\n\t" \
        "s_cbranch_scc1 LRWP%=\n\t" \
        "global_load_dwordx3 v[36:38], v10, s[44:45] sc0 sc1\n\t" \
        "global_load_dwordx3 v[40:42], v10, s[44:45] offset:768 sc0 sc1\n\t" \
        "s_bfe_u32 s62, s50, 0x70010\n\t" \
        "s_bfm_b64 s[60:61], s62, 0\n\t" \
        "s_cmp_eq_u32 s62, 64\n\t" \
        "s_cselect_b64 s[60:61], -1, s[60:61]\n\t" \
        "s_or_b32 s60, s60, 1\n\t" \
        "s_mov_b64 exec, s[60:61]\n\t" \
        "global_load_dword v44, v11, s[44:45] sc0 sc1\n\t" \
        "s_mov_b64 exec, -1\n\t" \
        "s_branch LLDP%=\n\t" \
        "\nLRWP%=:\n\t" \
        "s_lshl_b32 s62, %[cur], 12\n\t" \
        "s_add_u32 s44, s66, s62\n\t" \
        "s_addc_u32 s45, s67, 0\n\t" \
        "global_load_dwordx4 v[48:51], v8, s[44:45] offset:0 sc0 sc1\n\t" \
        "global_load_dwordx4 v[52:55], v8, s[44:45] offset:1024 sc0 sc1\n\t" \
        "global_load_dwordx4 v[56:59], v8, s[44:45] offset:2048 sc0 sc1\n\t" \
        "global_load_dwordx4 v[60:63], v8, s[44:45] offset:3072 sc0 sc1\n\t" \
        "\nLLDP%=:\n\t" \
        "s_min_u32 s62, %[nxt], 511\n\t" \
        "s_lshl_b32 s62, s62, 3\n\t" \
        "s_add_u32 s46, s68, s62\n\t" \
        "s_addc_u32 s47, s69, 0\n\t" \
        "s_load_dwordx2 s[48:49], s[46:47], 0x0 glc\n\t" \
        "s_mov_b32 s56, 0x03030303\n\t" \
        "s_mov_b32 s57, 0x7f7f7f7f\n\t" \
        "s_mov_b32 s58, 0x05010400\n\t" \
        "s_mov_b32 s59, 0x07030602\n\t" \
        "v_lshlrev_b32 v12, 5, %[lane]\n\t" \
        "v_add_u32 v12, %[lds], v12\n\t" \
        "v_mov_b32 v13, 0\n\t" \
        "v_mov_b32 v14, 1\n\t" \
        "v_mov_b32 v15, %[call]\n\t" \
        "s_add_u32 s62, %[nn], %[tb]\n\t" \
        "v_mov_b32 v16, s62\n\t" \
        "v_mov_b32 v164, %[lds]\n\t" \
        "v_add_u32 v165, 0x800, v164\n\t" \
        "s_waitcnt vmcnt(0)\n\t" \
        "\nLIA%=:\n\t" \
        "s_waitcnt vmcnt(10) lgkmcnt(0)\n\t" \
        "s_cmp_eq_u32 s49, %[call]\n\t" \
        "s_cselect_b32 s51, %[nxt], -1\n\t" \
        "s_cmp_lt_u32 s51, 512\n\t" \
        "s_cselect_b32 s51, 1, 0\n\t" \
        "s_cbranch_scc0 LNMA%=\n\t" \
        "s_mul_i32 s62, %[nxt], 5888\n\t" \
        "s_add_u32 s44, s64, s62\n\t" \
        "s_addc_u32 s45, s65, 0\n\t" \
        "global_load_dwordx4 v[68:71], v8, s[44:45] offset:0 sc0 sc1\n\t" \
        "global_load_dwordx4 v[72:75], v8, s[44:45] offset:1024 sc0 sc1\n\t" \
        "global_load_dwordx4 v[76:79], v8, s[44:45] offset:2048 sc0 sc1\n\t" \
        "global_load_dwordx4 v[80:83], v8, s[44:45] offset:3072 sc0 sc1\n\t" \
        "s_bitcmp1_b32 s48, 23\n\t" \
        "s_cbranch_scc1 LRWA%=\n\t" \
        "global_load_dwordx3 v[84:86], v10, s[44:45] sc0 sc1\n\t" \
        "global_load_dwordx3 v[88:90], v10, s[44:45] offset:768 sc0 sc1\n\t" \
        "s_bfe_u32 s62, s48, 0x70010\n\t" \
        "s_bfm_b64 s[60:61], s62, 0\n\t" \
        "s_cmp_eq_u32 s62, 64\n\t" \
        "s_cselect_b64 s[60:61], -1, s[60:61]\n\t" \
        "s_or_b32 s60, s60, 1\n\t" \
        "s_mov_b64 exec, s[60:61]\n\t" \
        "global_load_dword v92, v11, s[44:45] sc0 sc1\n\t" \
        "s_mov_b64 exec, -1\n\t" \
        "s_branch LLDA%=\n\t" \
        "\nLRWA%=:\n\t" \
        "s_lshl_b32 s62, %[nxt], 12\n\t" \
        "s_add_u32 s44, s66, s62\n\t" \
        "s_addc_u32 s45, s67, 0\n\t" \
        "global_load_dwordx4 v[96:99], v8, s[44:45] offset:0 sc0 sc1\n\t" \
        "global_load_dwordx4 v[100:103], v8, s[44:45] offset:1024 sc0 sc1\n\t" \
        "global_load_dwordx4 v[104:107], v8, s[44:45] offset:2048 sc0 sc1\n\t" \
        "global_load_dwordx4 v[108:111], v8, s[44:45] offset:3072 sc0 sc1\n\t" \
        "\nLLDA%=:\n\t" \
        "\nLNMA%=:\n\t" \
        "s_and_b32 s54, s50, 0x7f\n\t" \
        "s_mul_i32 s54, s54, 0x1010101\n\t" \
        "s_bfe_u32 s55, s50, 0x10007\n\t" \
        "s_mul_i32 s55, s55, 0x800080\n\t" \
        "s_bfe_u32 s74, s50, 0x70018\n\t" \
        "s_mul_i32 s74, s74, 0x1010101\n\t" \
        "s_lshr_b32 s75, s50, 31\n\t" \
        "s_mul_i32 s75, s75, 0x800080\n\t" \
        "s_bitcmp1_b32 s50, 23\n\t" \
        "s_cbranch_scc1 LDRA%=\n\t" \
        "s_bfe_u32 s63, s50, 0x70008\n\t" \
        "v_cmp_le_u32 vcc, s63, %[lane]\n\t" \
        "s_bfe_u32 s62, s50, 0x70010\n\t" \
        "s_bfm_b64 s[60:61], s62, 0\n\t" \
        "s_cmp_eq_u32 s62, 64\n\t" \
        "s_cselect_b64 s[60:61], -1, s[60:61]\n\t" \
        "v_cndmask_b32 v17, v164, v165, vcc\n\t" \
        "v_and_b32 v18, 0x7ff, v44\n\t" \
        "v_bfe_u32 v19, v44, 16, 11\n\t" \
        "v_lshrrev_b32 v160, 9, v44\n\t" \
        "v_lshrrev_b32 v161, 25, v44\n\t" \
        "v_add_u32 v18, v18, v17\n\t" \
        "v_add_u32 v19, v19, v17\n\t" \
        "v_and_b32 v160, 0x7c, v160\n\t" \
        "v_and_b32 v161, 0x7c, v161\n\t" \
        "s_mov_b64 exec, s[60:61]\n\t" \
        "ds_write_b8 v18, v160\n\t" \
        "ds_write_b8 v19, v161\n\t" \
        "s_mov_b64 exec, -1\n\t" \
        "ds_read_b128 v[112:115], v12 offset:0\n\t" \
        "ds_read_b128 v[116:119], v12 offset:16\n\t" \
        "ds_read_b128 v[120:123], v12 offset:2048\n\t" \
        "ds_read_b128 v[124:127], v12 offset:2064\n\t" \
        "s_mov_b64 exec, s[60:61]\n\t" \
        "ds_write_b8 v18, v13\n\t" \
        "ds_write_b8 v19, v13\n\t" \
        "s_mov_b64 exec, -1\n\t" \
        "v_lshrrev_b32 v129, 2, v36\n\t" \
        "v_lshrrev_b32 v130, 4, v36\n\t" \
        "v_lshrrev_b32 v131, 6, v36\n\t" \
        "v_lshrrev_b32 v133, 2, v37\n\t" \
        "v_lshrrev_b32 v134, 4, v37\n\t" \
        "v_lshrrev_b32 v135, 6, v37\n\t" \
        "v_lshrrev_b32 v137, 2, v40\n\t" \
        "v_lshrrev_b32 v138, 4, v40\n\t" \
        "v_lshrrev_b32 v139, 6, v40\n\t" \
        "v_lshrrev_b32 v141, 2, v41\n\t" \
        "v_lshrrev_b32 v142, 4, v41\n\t" \
        "v_lshrrev_b32 v143, 6, v41\n\t" \
        "v_lshlrev_b32 v145, 1, v38\n\t" \
        "v_lshlrev_b32 v146, 2, v38\n\t" \
        "v_lshlrev_b32 v147, 3, v38\n\t" \
        "v_lshlrev_b32 v148, 4, v38\n\t" \
        "v_lshlrev_b32 v149, 5, v38\n\t" \
        "v_lshlrev_b32 v150, 6, v38\n\t" \
        "v_lshlrev_b32 v151, 7, v38\n\t" \
        "v_lshlrev_b32 v153, 1, v42\n\t" \
        "v_lshlrev_b32 v154, 2, v42\n\t" \
        "v_lshlrev_b32 v155, 3, v42\n\t" \
        "v_lshlrev_b32 v156, 4, v42\n\t" \
        "v_lshlrev_b32 v157, 5, v42\n\t" \
        "v_lshlrev_b32 v158, 6, v42\n\t" \
        "v_lshlrev_b32 v159, 7, v42\n\t" \
        "s_waitcnt lgkmcnt(0)\n\t" \
        "v_and_or_b32 v112, v36, s56, v112\n\t" \
        "v_and_or_b32 v113, v129, s56, v113\n\t" \
        "v_and_or_b32 v114, v130, s56, v114\n\t" \
        "v_and_or_b32 v115, v131, s56, v115\n\t" \
        "v_and_or_b32 v116, v37, s56, v116\n\t" \
        "v_and_or_b32 v117, v133, s56, v117\n\t" \
        "v_and_or_b32 v118, v134, s56, v118\n\t" \
        "v_and_or_b32 v119, v135, s56, v119\n\t" \
        "v_and_or_b32 v120, v40, s56, v120\n\t" \
        "v_and_or_b32 v121, v137, s56, v121\n\t" \
        "v_and_or_b32 v122, v138, s56, v122\n\t" \
        "v_and_or_b32 v123, v139, s56, v123\n\t" \
        "v_and_or_b32 v124, v41, s56, v124\n\t" \
        "v_and_or_b32 v125, v141, s56, v125\n\t" \
        "v_and_or_b32 v126, v142, s56, v126\n\t" \
        "v_and_or_b32 v127, v143, s56, v127\n\t" \
        "v_add_u32 v112, s54, v112\n\t" \
        "v_add_u32 v113, s54, v113\n\t" \
        "v_add_u32 v114, s54, v114\n\t" \
        "v_add_u32 v115, s54, v115\n\t" \
        "v_add_u32 v116, s54, v116\n\t" \
        "v_add_u32 v117, s54, v117\n\t" \
        "v_add_u32 v118, s54, v118\n\t" \
        "v_add_u32 v119, s54, v119\n\t" \
        "v_add_u32 v120, s74, v120\n\t" \
        "v_add_u32 v121, s74, v121\n\t" \
        "v_add_u32 v122, s74, v122\n\t" \
        "v_add_u32 v123, s74, v123\n\t" \
        "v_add_u32 v124, s74, v124\n\t" \
        "v_add_u32 v125, s74, v125\n\t" \
        "v_add_u32 v126, s74, v126\n\t" \
        "v_add_u32 v127, s74, v127\n\t" \
        "v_bfi_b32 v112, s57, v112, v38\n\t" \
        "v_bfi_b32 v113, s57, v113, v145\n\t" \
        "v_bfi_b32 v114, s57, v114, v146\n\t" \
        "v_bfi_b32 v115, s57, v115, v147\n\t" \
        "v_bfi_b32 v116, s57, v116, v148\n\t" \
        "v_bfi_b32 v117, s57, v117, v149\n\t" \
        "v_bfi_b32 v118, s57, v118, v150\n\t" \
        "v_bfi_b32 v119, s57, v119, v151\n\t" \
        "v_bfi_b32 v120, s57, v120, v42\n\t" \
        "v_bfi_b32 v121, s57, v121, v153\n\t" \
        "v_bfi_b32 v122, s57, v122, v154\n\t" \
        "v_bfi_b32 v123, s57, v123, v155\n\t" \
        "v_bfi_b32 v124, s57, v124, v156\n\t" \
        "v_bfi_b32 v125, s57, v125, v157\n\t" \
        "v_bfi_b32 v126, s57, v126, v158\n\t" \
        "v_bfi_b32 v127, s57, v127, v159\n\t" \
        "\nLDPA%=:\n\t" \
        "v_perm_b32 v128, v112, v20, s58\n\t" \
        "v_perm_b32 v129, v112, v20, s59\n\t" \
        "v_perm_b32 v130, v113, v21, s58\n\t" \
        "v_perm_b32 v131, v113, v21, s59\n\t" \
        "v_perm_b32 v132, v114, v22, s58\n\t" \
        "v_perm_b32 v133, v114, v22, s59\n\t" \
        "v_perm_b32 v134, v115, v23, s58\n\t" \
        "v_perm_b32 v135, v115, v23, s59\n\t" \
        "v_perm_b32 v136, v116, v24, s58\n\t" \
        "v_perm_b32 v137, v116, v24, s59\n\t" \
        "v_perm_b32 v138, v117, v25, s58\n\t" \
        "v_perm_b32 v139, v117, v25, s59\n\t" \
        "v_perm_b32 v140, v118, v26, s58\n\t" \
        "v_perm_b32 v141, v118, v26, s59\n\t" \
        "v_perm_b32 v142, v119, v27, s58\n\t" \
        "v_perm_b32 v143, v119, v27, s59\n\t" \
        "v_perm_b32 v144, v120, v28, s58\n\t" \
        "v_perm_b32 v145, v120, v28, s59\n\t" \
        "v_perm_b32 v146, v121, v29, s58\n\t" \
        "v_perm_b32 v147, v121, v29, s59\n\t" \
        "v_perm_b32 v148, v122, v30, s58\n\t" \
        "v_perm_b32 v149, v122, v30, s59\n\t" \
        "v_perm_b32 v150, v123, v31, s58\n\t" \
        "v_perm_b32 v151, v123, v31, s59\n\t" \
        "v_perm_b32 v152, v124, v32, s58\n\t" \
        "v_perm_b32 v153, v124, v32, s59\n\t" \
        "v_perm_b32 v154, v125, v33, s58\n\t" \
        "v_perm_b32 v155, v125, v33, s59\n\t" \
        "v_perm_b32 v156, v126, v34, s58\n\t" \
        "v_perm_b32 v157, v126, v34, s59\n\t" \
        "v_perm_b32 v158, v127, v35, s58\n\t" \
        "v_perm_b32 v159, v127, v35, s59\n\t" \
        "v_pk_sub_u16 v128, v128, s55\n\t" \
        "v_pk_sub_u16 v129, v129, s55\n\t" \
        "v_pk_sub_u16 v130, v130, s55\n\t" \
        "v_pk_sub_u16 v131, v131, s55\n\t" \
        "v_pk_sub_u16 v132, v132, s55\n\t" \
        "v_pk_sub_u16 v133, v133, s55\n\t" \
        "v_pk_sub_u16 v134, v134, s55\n\t" \
        "v_pk_sub_u16 v135, v135, s55\n\t" \
        "v_pk_sub_u16 v136, v136, s55\n\t" \
        "v_pk_sub_u16 v137, v137, s55\n\t" \
        "v_pk_sub_u16 v138, v138, s55\n\t" \
        "v_pk_sub_u16 v139, v139, s55\n\t" \
        "v_pk_sub_u16 v140, v140, s55\n\t" \
        "v_pk_sub_u16 v141, v141, s55\n\t" \
        "v_pk_sub_u16 v142, v142, s55\n\t" \
        "v_pk_sub_u16 v143, v143, s55\n\t" \
        "v_pk_sub_u16 v144, v144, s75\n\t" \
        "v_pk_sub_u16 v145, v145, s75\n\t" \
        "v_pk_sub_u16 v146, v146, s75\n\t" \
        "v_pk_sub_u16 v147, v147, s75\n\t" \
        "v_pk_sub_u16 v148, v148, s75\n\t" \
        "v_pk_sub_u16 v149, v149, s75\n\t" \
        "v_pk_sub_u16 v150, v150, s75\n\t" \
        "v_pk_sub_u16 v151, v151, s75\n\t" \
        "v_pk_sub_u16 v152, v152, s75\n\t" \
        "v_pk_sub_u16 v153, v153, s75\n\t" \
        "v_pk_sub_u16 v154, v154, s75\n\t" \
        "v_pk_sub_u16 v155, v155, s75\n\t" \
        "v_pk_sub_u16 v156, v156, s75\n\t" \
        "v_pk_sub_u16 v157, v157, s75\n\t" \
        "v_pk_sub_u16 v158, v158, s75\n\t" \
        "v_pk_sub_u16 v159, v159, s75\n\t" \
        "s_cmp_eq_u32 s51, 0\n\t" \
        "s_cbranch_scc0 LW7A%=\n\t" \
        "s_waitcnt vmcnt(0)\n\t" \
        "\nLW7A%=:\n\t" \
        "s_waitcnt vmcnt(7)\n\t" \
        "s_mov_b64 exec, 1\n\t" \
        "global_store_dword v13, v15, s[52:53] sc0 sc1\n\t" \
        "s_mov_b64 exec, -1\n\t" \
        "v_readfirstlane_b32 s62, v16\n\t" \
        "s_sub_u32 s62, s62, %[tb]\n\t" \
        "s_cmp_lt_u32 %[nxt], 512\n\t" \
        "s_cselect_b32 %[nn], s62, %[nxt]\n\t" \
        "s_mov_b32 s50, s48\n\t" \
        "s_min_u32 s62, %[nn], 511\n\t" \
        "s_lshl_b32 s62, s62, 3\n\t" \
        "s_add_u32 s46, s68, s62\n\t" \
        "s_addc_u32 s47, s69, 0\n\t" \
        "s_load_dwordx2 s[48:49], s[46:47], 0x0 glc\n\t" \
        "s_cmp_lt_u32 %[nn], 512\n\t" \
        "s_cselect_b64 s[60:61], %[tk], %[dum]\n\t" \
        "s_mov_b64 exec, 1\n\t" \
        "global_atomic_add v16, v13, v14, s[60:61] sc0\n\t" \
        "s_mov_b64 exec, -1\n\t" \
        "s_bfe_u32 s62, %[cur], 0x10001\n\t" \
        "s_lshl_b32 s62, s62, 8\n\t" \
        "s_lshr_b32 s63, %[cur], 2\n\t" \
        "s_lshl_b32 s63, s63, 1\n\t" \
        "s_add_u32 s62, s62, s63\n\t" \
        "s_and_b32 s63, %[cur], 1\n\t" \
        "s_add_u32 s62, s62, s63\n\t" \
        "s_lshl_b32 s63, s62, 13\n\t" \
        "s_bitcmp1_b32 %[cur], 1\n\t" \
        "s_cselect_b32 s44, %[sk], 0\n\t" \
        "s_add_u32 s63, s63, s44\n\t" \
        "s_add_u32 s44, s70, s63\n\t" \
        "s_addc_u32 s45, s71, 0\n\t" \
        "s_lshl_b32 s62, s62, 2\n\t" \
        "s_add_u32 s52, s72, s62\n\t" \
        "s_addc_u32 s53, s73, 0\n\t" \
        "global_store_dwordx4 v8, v[128:131], s[44:45] offset:0 sc0 sc1\n\t" \
        "global_store_dwordx4 v8, v[132:135], s[44:45] offset:1024 sc0 sc1\n\t" \
        "global_store_dwordx4 v8, v[136:139], s[44:45] offset:2048 sc0 sc1\n\t" \
        "global_store_dwordx4 v8, v[140:143], s[44:45] offset:3072 sc0 sc1\n\t" \
        "global_store_dwordx4 v9, v[144:147], s[44:45] offset:0 sc0 sc1\n\t" \
        "global_store_dwordx4 v9, v[148:151], s[44:45] offset:1024 sc0 sc1\n\t" \
        "global_store_dwordx4 v9, v[152:155], s[44:45] offset:2048 sc0 sc1\n\t" \
        "global_store_dwordx4 v9, v[156:159], s[44:45] offset:3072 sc0 sc1\n\t" \
        "s_mov_b32 %[cur], %[nxt]\n\t" \
        "s_mov_b32 %[nxt], %[nn]\n\t" \
        "s_cmp_eq_u32 s51, 0\n\t" \
        "s_cbranch_scc1 LEX%=\n\t" \
        "\nLIB%=:\n\t" \
        "s_waitcnt vmcnt(10) lgkmcnt(0)\n\t" \
        "s_cmp_eq_u32 s49, %[call]\n\t" \
        "s_cselect_b32 s51, %[nxt], -1\n\t" \
        "s_cmp_lt_u32 s51, 512\n\t" \
        "s_cselect_b32 s51, 1, 0\n\t" \
        "s_cbranch_scc0 LNMB%=\n\t" \
        "s_mul_i32 s62, %[nxt], 5888\n\t" \
        "s_add_u32 s44, s64, s62\n\t" \
        "s_addc_u32 s45, s65, 0\n\t" \
        "global_load_dwordx4 v[20:23], v8, s[44:45] offset:0 sc0 sc1\n\t" \
        "global_load_dwordx4 v[24:27], v8, s[44:45] offset:1024 sc0 sc1\n\t" \
        "global_load_dwordx4 v[28:31], v8, s[44:45] offset:2048 sc0 sc1\n\t" \
        "global_load_dwordx4 v[32:35], v8, s[44:45] offset:3072 sc0 sc1\n\t" \
        "s_bitcmp1_b32 s48, 23\n\t" \
        "s_cbranch_scc1 LRWB%=\n\t" \
        "global_load_dwordx3 v[36:38], v10, s[44:45] sc0 sc1\n\t" \
        "global_load_dwordx3 v[40:42], v10, s[44:45] offset:768 sc0 sc1\n\t" \
        "s_bfe_u32 s62, s48, 0x70010\n\t" \
        "s_bfm_b64 s[60:61], s62, 0\n\t" \
        "s_cmp_eq_u32 s62, 64\n\t" \
        "s_cselect_b64 s[60:61], -1, s[60:61]\n\t" \
        "s_or_b32 s60, s60, 1\n\t" \
        "s_mov_b64 exec, s[60:61]\n\t" \
        "global_load_dword v44, v11, s[44:45] sc0 sc1\n\t" \
        "s_mov_b64 exec, -1\n\t" \
        "s_branch LLDB%=\n\t" \
        "\nLRWB%=:\n\t" \
        "s_lshl_b32 s62, %[nxt], 12\n\t" \
        "s_add_u32 s44, s66, s62\n\t" \
        "s_addc_u32 s45, s67, 0\n\t" \
        "global_load_dwordx4 v[48:51], v8, s[44:45] offset:0 sc0 sc1\n\t" \
        "global_load_dwordx4 v[52:55], v8, s[44:45] offset:1024 sc0 sc1\n\t" \
        "global_load_dwordx4 v[56:59], v8, s[44:45] offset:2048 sc0 sc1\n\t" \
        "global_load_dwordx4 v[60:63], v8, s[44:45] offset:3072 sc0 sc1\n\t" \
        "\nLLDB%=:\n\t" \
        "\nLNMB%=:\n\t" \
        "s_and_b32 s54, s50, 0x7f\n\t" \
        "s_mul_i32 s54, s54, 0x1010101\n\t" \
        "s_bfe_u32 s55, s50, 0x10007\n\t" \
        "s_mul_i32 s55, s55, 0x800080\n\t" \
        "s_bfe_u32 s74, s50, 0x70018\n\t" \
        "s_mul_i32 s74, s74, 0x1010101\n\t" \
        "s_lshr_b32 s75, s50, 31\n\t" \
        "s_mul_i32 s75, s75, 0x800080\n\t" \
        "s_bitcmp1_b32 s50, 23\n\t" \
        "s_cbranch_scc1 LDRB%=\n\t" \
        "s_bfe_u32 s63, s50, 0x70008\n\t" \
        "v_cmp_le_u32 vcc, s63, %[lane]\n\t" \
        "s_bfe_u32 s62, s50, 0x70010\n\t" \
        "s_bfm_b64 s[60:61], s62, 0\n\t" \
        "s_cmp_eq_u32 s62, 64\n\t" \
        "s_cselect_b64 s[60:61], -1, s[60:61]\n\t" \
        "v_cndmask_b32 v17, v164, v165, vcc\n\t" \
        "v_and_b32 v18, 0x7ff, v92\n\t" \
        "v_bfe_u32 v19, v92, 16, 11\n\t" \
        "v_lshrrev_b32 v160, 9, v92\n\t" \
        "v_lshrrev_b32 v161, 25, v92\n\t" \
        "v_add_u32 v18, v18, v17\n\t" \
        "v_add_u32 v19, v19, v17\n\t" \
        "v_and_b32 v160, 0x7c, v160\n\t" \
        "v_and_b32 v161, 0x7c, v161\n\t" \
        "s_mov_b64 exec, s[60:61]\n\t" \
        "ds_write_b8 v18, v160\n\t" \
        "ds_write_b8 v19, v161\n\t" \
        "s_mov_b64 exec, -1\n\t" \
        "ds_read_b128 v[112:115], v12 offset:0\n\t" \
        "ds_read_b128 v[116:119], v12 offset:16\n\t" \
        "ds_read_b128 v[120:123], v12 offset:2048\n\t" \
        "ds_read_b128 v[124:127], v12 offset:2064\n\t" \
        "s_mov_b64 exec, s[60:61]\n\t" \
        "ds_write_b8 v18, v13\n\t" \
        "ds_write_b8 v19, v13\n\t" \
        "s_mov_b64 exec, -1\n\t" \
        "v_lshrrev_b32 v129, 2, v84\n\t" \
        "v_lshrrev_b32 v130, 4, v84\n\t" \
        "v_lshrrev_b32 v131, 6, v84\n\t" \
        "v_lshrrev_b32 v133, 2, v85\n\t" \
        "v_lshrrev_b32 v134, 4, v85\n\t" \
        "v_lshrrev_b32 v135, 6, v85\n\t" \
        "v_lshrrev_b32 v137, 2, v88\n\t" \
        "v_lshrrev_b32 v138, 4, v88\n\t" \
        "v_lshrrev_b32 v139, 6, v88\n\t" \
        "v_lshrrev_b32 v141, 2, v89\n\t" \
        "v_lshrrev_b32 v142, 4, v89\n\t" \
        "v_lshrrev_b32 v143, 6, v89\n\t" \
        "v_lshlrev_b32 v145, 1, v86\n\t" \
        "v_lshlrev_b32 v146, 2, v86\n\t" \
        "v_lshlrev_b32 v147, 3, v86\n\t" \
        "v_lshlrev_b32 v148, 4, v86\n\t" \
        "v_lshlrev_b32 v149, 5, v86\n\t" \
        "v_lshlrev_b32 v150, 6, v86\n\t" \
        "v_lshlrev_b32 v151, 7, v86\n\t" \
        "v_lshlrev_b32 v153, 1, v90\n\t" \
        "v_lshlrev_b32 v154, 2, v90\n\t" \
        "v_lshlrev_b32 v155, 3, v90\n\t" \
        "v_lshlrev_b32 v156, 4, v90\n\t" \
        "v_lshlrev_b32 v157, 5, v90\n\t" \
        "v_lshlrev_b32 v158, 6, v90\n\t" \
        "v_lshlrev_b32 v159, 7, v90\n\t" \
        "s_waitcnt lgkmcnt(0)\n\t" \
        "v_and_or_b32 v112, v84, s56, v112\n\t" \
        "v_and_or_b32 v113, v129, s56, v113\n\t" \
        "v_and_or_b32 v114, v130, s56, v114\n\t" \
        "v_and_or_b32 v115, v131, s56, v115\n\t" \
        "v_and_or_b32 v116, v85, s56, v116\n\t" \
        "v_and_or_b32 v117, v133, s56, v117\n\t" \
        "v_and_or_b32 v118, v134, s56, v118\n\t" \
        "v_and_or_b32 v119, v135, s56, v119\n\t" \
        "v_and_or_b32 v120, v88, s56, v120\n\t" \
        "v_and_or_b32 v121, v137, s56, v121\n\t" \
        "v_and_or_b32 v122, v138, s56, v122\n\t" \
        "v_and_or_b32 v123, v139, s56, v123\n\t" \
        "v_and_or_b32 v124, v89, s56, v124\n\t" \
        "v_and_or_b32 v125, v141, s56, v125\n\t" \
        "v_and_or_b32 v126, v142, s56, v126\n\t" \
        "v_and_or_b32 v127, v143, s56, v127\n\t" \
        "v_add_u32 v112, s54, v112\n\t" \
        "v_add_u32 v113, s54, v113\n\t" \
        "v_add_u32 v114, s54, v114\n\t" \
        "v_add_u32 v115, s54, v115\n\t" \
        "v_add_u32 v116, s54, v116\n\t" \
        "v_add_u32 v117, s54, v117\n\t" \
        "v_add_u32 v118, s54, v118\n\t" \
        "v_add_u32 v119, s54, v119\n\t" \
        "v_add_u32 v120, s74, v120\n\t" \
        "v_add_u32 v121, s74, v121\n\t" \
        "v_add_u32 v122, s74, v122\n\t" \
        "v_add_u32 v123, s74, v123\n\t" \
        "v_add_u32 v124, s74, v124\n\t" \
        "v_add_u32 v125, s74, v125\n\t" \
        "v_add_u32 v126, s74, v126\n\t" \
        "v_add_u32 v127, s74, v127\n\t" \
        "v_bfi_b32 v112, s57, v112, v86\n\t" \
        "v_bfi_b32 v113, s57, v113, v145\n\t" \
        "v_bfi_b32 v114, s57, v114, v146\n\t" \
        "v_bfi_b32 v115, s57, v115, v147\n\t" \
        "v_bfi_b32 v116, s57, v116, v148\n\t" \
        "v_bfi_b32 v117, s57, v117, v149\n\t" \
        "v_bfi_b32 v118, s57, v118, v150\n\t" \
        "v_bfi_b32 v119, s57, v119, v151\n\t" \
        "v_bfi_b32 v120, s57, v120, v90\n\t" \
        "v_bfi_b32 v121, s57, v121, v153\n\t" \
        "v_bfi_b32 v122, s57, v122, v154\n\t" \
        "v_bfi_b32 v123, s57, v123, v155\n\t" \
        "v_bfi_b32 v124, s57, v124, v156\n\t" \
        "v_bfi_b32 v125, s57, v125, v157\n\t" \
        "v_bfi_b32 v126, s57, v126, v158\n\t" \
        "v_bfi_b32 v127, s57, v127, v159\n\t" \
        "\nLDPB%=:\n\t" \
        "v_perm_b32 v128, v112, v68, s58\n\t" \
        "v_perm_b32 v129, v112, v68, s59\n\t" \
        "v_perm_b32 v130, v113, v69, s58\n\t" \
        "v_perm_b32 v131, v113, v69, s59\n\t" \
        "v_perm_b32 v132, v114, v70, s58\n\t" \
        "v_perm_b32 v133, v114, v70, s59\n\t" \
        "v_perm_b32 v134, v115, v71, s58\n\t" \
        "v_perm_b32 v135, v115, v71, s59\n\t" \
        "v_perm_b32 v136, v116, v72, s58\n\t" \
        "v_perm_b32 v137, v116, v72, s59\n\t" \
        "v_perm_b32 v138, v117, v73, s58\n\t" \
        "v_perm_b32 v139, v117, v73, s59\n\t" \
        "v_perm_b32 v140, v118, v74, s58\n\t" \
        "v_perm_b32 v141, v118, v74, s59\n\t" \
        "v_perm_b32 v142, v119, v75, s58\n\t" \
        "v_perm_b32 v143, v119, v75, s59\n\t" \
        "v_perm_b32 v144, v120, v76, s58\n\t" \
        "v_perm_b32 v145, v120, v76, s59\n\t" \
        "v_perm_b32 v146, v121, v77, s58\n\t" \
        "v_perm_b32 v147, v121, v77, s59\n\t" \
        "v_perm_b32 v148, v122, v78, s58\n\t" \
        "v_perm_b32 v149, v122, v78, s59\n\t" \
        "v_perm_b32 v150, v123, v79, s58\n\t" \
        "v_perm_b32 v151, v123, v79, s59\n\t" \
        "v_perm_b32 v152, v124, v80, s58\n\t" \
        "v_perm_b32 v153, v124, v80, s59\n\t" \
        "v_perm_b32 v154, v125, v81, s58\n\t" \
        "v_perm_b32 v155, v125, v81, s59\n\t" \
        "v_perm_b32 v156, v126, v82, s58\n\t" \
        "v_perm_b32 v157, v126, v82, s59\n\t" \
        "v_perm_b32 v158, v127, v83, s58\n\t" \
        "v_perm_b32 v159, v127, v83, s59\n\t" \
        "v_pk_sub_u16 v128, v128, s55\n\t" \
        "v_pk_sub_u16 v129, v129, s55\n\t" \
        "v_pk_sub_u16 v130, v130, s55\n\t" \
        "v_pk_sub_u16 v131, v131, s55\n\t" \
        "v_pk_sub_u16 v132, v132, s55\n\t" \
        "v_pk_sub_u16 v133, v133, s55\n\t" \
        "v_pk_sub_u16 v134, v134, s55\n\t" \
        "v_pk_sub_u16 v135, v135, s55\n\t" \
        "v_pk_sub_u16 v136, v136, s55\n\t" \
        "v_pk_sub_u16 v137, v137, s55\n\t" \
        "v_pk_sub_u16 v138, v138, s55\n\t" \
        "v_pk_sub_u16 v139, v139, s55\n\t" \
        "v_pk_sub_u16 v140, v140, s55\n\t" \
        "v_pk_sub_u16 v141, v141, s55\n\t" \
        "v_pk_sub_u16 v142, v142, s55\n\t" \
        "v_pk_sub_u16 v143, v143, s55\n\t" \
        "v_pk_sub_u16 v144, v144, s75\n\t" \
        "v_pk_sub_u16 v145, v145, s75\n\t" \
        "v_pk_sub_u16 v146, v146, s75\n\t" \
        "v_pk_sub_u16 v147, v147, s75\n\t" \
        "v_pk_sub_u16 v148, v148, s75\n\t" \
        "v_pk_sub_u16 v149, v149, s75\n\t" \
        "v_pk_sub_u16 v150, v150, s75\n\t" \
        "v_pk_sub_u16 v151, v151, s75\n\t" \
        "v_pk_sub_u16 v152, v152, s75\n\t" \
        "v_pk_sub_u16 v153, v153, s75\n\t" \
        "v_pk_sub_u16 v154, v154, s75\n\t" \
        "v_pk_sub_u16 v155, v155, s75\n\t" \
        "v_pk_sub_u16 v156, v156, s75\n\t" \
        "v_pk_sub_u16 v157, v157, s75\n\t" \
        "v_pk_sub_u16 v158, v158, s75\n\t" \
        "v_pk_sub_u16 v159, v159, s75\n\t" \
        "s_cmp_eq_u32 s51, 0\n\t" \
        "s_cbranch_scc0 LW7B%=\n\t" \
        "s_waitcnt vmcnt(0)\n\t" \
        "\nLW7B%=:\n\t" \
        "s_waitcnt vmcnt(7)\n\t" \
        "s_mov_b64 exec, 1\n\t" \
        "global_store_dword v13, v15, s[52:53] sc0 sc1\n\t" \
        "s_mov_b64 exec, -1\n\t" \
        "v_readfirstlane_b32 s62, v16\n\t" \
        "s_sub_u32 s62, s62, %[tb]\n\t" \
        "s_cmp_lt_u32 %[nxt], 512\n\t" \
        "s_cselect_b32 %[nn], s62, %[nxt]\n\t" \
        "s_mov_b32 s50, s48\n\t" \
        "s_min_u32 s62, %[nn], 511\n\t" \
        "s_lshl_b32 s62, s62, 3\n\t" \
        "s_add_u32 s46, s68, s62\n\t" \
        "s_addc_u32 s47, s69, 0\n\t" \
        "s_load_dwordx2 s[48:49], s[46:47], 0x0 glc\n\t" \
        "s_cmp_lt_u32 %[nn], 512\n\t" \
        "s_cselect_b64 s[60:61], %[tk], %[dum]\n\t" \
        "s_mov_b64 exec, 1\n\t" \
        "global_atomic_add v16, v13, v14, s[60:61] sc0\n\t" \
        "s_mov_b64 exec, -1\n\t" \
        "s_bfe_u32 s62, %[cur], 0x10001\n\t" \
        "s_lshl_b32 s62, s62, 8\n\t" \
        "s_lshr_b32 s63, %[cur], 2\n\t" \
        "s_lshl_b32 s63, s63, 1\n\t" \
        "s_add_u32 s62, s62, s63\n\t" \
        "s_and_b32 s63, %[cur], 1\n\t" \
        "s_add_u32 s62, s62, s63\n\t" \
        "s_lshl_b32 s63, s62, 13\n\t" \
        "s_bitcmp1_b32 %[cur], 1\n\t" \
        "s_cselect_b32 s44, %[sk], 0\n\t" \
        "s_add_u32 s63, s63, s44\n\t" \
        "s_add_u32 s44, s70, s63\n\t" \
        "s_addc_u32 s45, s71, 0\n\t" \
        "s_lshl_b32 s62, s62, 2\n\t" \
        "s_add_u32 s52, s72, s62\n\t" \
        "s_addc_u32 s53, s73, 0\n\t" \
        "global_store_dwordx4 v8, v[128:131], s[44:45] offset:0 sc0 sc1\n\t" \
        "global_store_dwordx4 v8, v[132:135], s[44:45] offset:1024 sc0 sc1\n\t" \
        "global_store_dwordx4 v8, v[136:139], s[44:45] offset:2048 sc0 sc1\n\t" \
        "global_store_dwordx4 v8, v[140:143], s[44:45] offset:3072 sc0 sc1\n\t" \
        "global_store_dwordx4 v9, v[144:147], s[44:45] offset:0 sc0 sc1\n\t" \
        "global_store_dwordx4 v9, v[148:151], s[44:45] offset:1024 sc0 sc1\n\t" \
        "global_store_dwordx4 v9, v[152:155], s[44:45] offset:2048 sc0 sc1\n\t" \
        "global_store_dwordx4 v9, v[156:159], s[44:45] offset:3072 sc0 sc1\n\t" \
        "s_mov_b32 %[cur], %[nxt]\n\t" \
        "s_mov_b32 %[nxt], %[nn]\n\t" \
        "s_cmp_eq_u32 s51, 0\n\t" \
        "s_cbranch_scc0 LIA%=\n\t" \
        "s_branch LEX%=\n\t" \
        "\nLDRA%=:\n\t" \
        "v_mov_b32 v112, v48\n\t" \
        "v_mov_b32 v113, v49\n\t" \
        "v_mov_b32 v114, v50\n\t" \
        "v_mov_b32 v115, v51\n\t" \
        "v_mov_b32 v116, v52\n\t" \
        "v_mov_b32 v117, v53\n\t" \
        "v_mov_b32 v118, v54\n\t" \
        "v_mov_b32 v119, v55\n\t" \
        "v_mov_b32 v120, v56\n\t" \
        "v_mov_b32 v121, v57\n\t" \
        "v_mov_b32 v122, v58\n\t" \
        "v_mov_b32 v123, v59\n\t" \
        "v_mov_b32 v124, v60\n\t" \
        "v_mov_b32 v125, v61\n\t" \
        "v_mov_b32 v126, v62\n\t" \
        "v_mov_b32 v127, v63\n\t" \
        "s_branch LDPA%=\n\t" \
        "\nLDRB%=:\n\t" \
        "v_mov_b32 v112, v96\n\t" \
        "v_mov_b32 v113, v97\n\t" \
        "v_mov_b32 v114, v98\n\t" \
        "v_mov_b32 v115, v99\n\t" \
        "v_mov_b32 v116, v100\n\t" \
        "v_mov_b32 v117, v101\n\t" \
        "v_mov_b32 v118, v102\n\t" \
        "v_mov_b32 v119, v103\n\t" \
        "v_mov_b32 v120, v104\n\t" \
        "v_mov_b32 v121, v105\n\t" \
        "v_mov_b32 v122, v106\n\t" \
        "v_mov_b32 v123, v107\n\t" \
        "v_mov_b32 v124, v108\n\t" \
        "v_mov_b32 v125, v109\n\t" \
        "v_mov_b32 v126, v110\n\t" \
        "v_mov_b32 v127, v111\n\t" \
        "s_branch LDPB%=\n\t" \
        "\nLEX%=:\n\t" \
        "s_waitcnt vmcnt(0) lgkmcnt(0)\n\t" \
        "s_mov_b64 exec, 1\n\t" \
        "global_store_dword v13, v15, s[52:53] sc0 sc1\n\t" \
        "s_mov_b64 exec, -1\n\t" \
        "v_readfirstlane_b32 s62, v16\n\t" \
        "s_sub_u32 s62, s62, %[tb]\n\t" \
        "s_cmp_lt_u32 %[nxt], 512\n\t" \
        "s_cselect_b32 %[nn], s62, %[nxt]\n\t"
#define RELU \
        "v_max_f32 v0, 0, v0\n\t" \
        "v_max_f32 v1, 0, v1\n\t" \
        "v_max_f32 v2, 0, v2\n\t" \
        "v_max_f32 v3, 0, v3\n\t" \
        "v_max_f32 v4, 0, v4\n\t" \
        "v_max_f32 v5, 0, v5\n\t" \
        "v_max_f32 v6, 0, v6\n\t" \
        "v_max_f32 v7, 0, v7\n\t" \
        "v_max_f32 v8, 0, v8\n\t" \
        "v_max_f32 v9, 0, v9\n\t" \
        "v_max_f32 v10, 0, v10\n\t" \
        "v_max_f32 v11, 0, v11\n\t" \
        "v_max_f32 v12, 0, v12\n\t" \
        "v_max_f32 v13, 0, v13\n\t" \
        "v_max_f32 v14, 0, v14\n\t" \
        "v_max_f32 v15, 0, v15\n\t" \
        "v_max_f32 v16, 0, v16\n\t" \
        "v_max_f32 v17, 0, v17\n\t" \
        "v_max_f32 v18, 0, v18\n\t" \
        "v_max_f32 v19, 0, v19\n\t" \
        "v_max_f32 v20, 0, v20\n\t" \
        "v_max_f32 v21, 0, v21\n\t" \
        "v_max_f32 v22, 0, v22\n\t" \
        "v_max_f32 v23, 0, v23\n\t" \
        "v_max_f32 v24, 0, v24\n\t" \
        "v_max_f32 v25, 0, v25\n\t" \
        "v_max_f32 v26, 0, v26\n\t" \
        "v_max_f32 v27, 0, v27\n\t" \
        "v_max_f32 v28, 0, v28\n\t" \
        "v_max_f32 v29, 0, v29\n\t" \
        "v_max_f32 v30, 0, v30\n\t" \
        "v_max_f32 v31, 0, v31\n\t" \
        "v_max_f32 v32, 0, v32\n\t" \
        "v_max_f32 v33, 0, v33\n\t" \
        "v_max_f32 v34, 0, v34\n\t" \
        "v_max_f32 v35, 0, v35\n\t" \
        "v_max_f32 v36, 0, v36\n\t" \
        "v_max_f32 v37, 0, v37\n\t" \
        "v_max_f32 v38, 0, v38\n\t" \
        "v_max_f32 v39, 0, v39\n\t" \
        "v_max_f32 v40, 0, v40\n\t" \
        "v_max_f32 v41, 0, v41\n\t" \
        "v_max_f32 v42, 0, v42\n\t" \
        "v_max_f32 v43, 0, v43\n\t" \
        "v_max_f32 v44, 0, v44\n\t" \
        "v_max_f32 v45, 0, v45\n\t" \
        "v_max_f32 v46, 0, v46\n\t" \
        "v_max_f32 v47, 0, v47\n\t" \
        "v_max_f32 v48, 0, v48\n\t" \
        "v_max_f32 v49, 0, v49\n\t" \
        "v_max_f32 v50, 0, v50\n\t" \
        "v_max_f32 v51, 0, v51\n\t" \
        "v_max_f32 v52, 0, v52\n\t" \
        "v_max_f32 v53, 0, v53\n\t" \
        "v_max_f32 v54, 0, v54\n\t" \
        "v_max_f32 v55, 0, v55\n\t" \
        "v_max_f32 v56, 0, v56\n\t" \
        "v_max_f32 v57, 0, v57\n\t" \
        "v_max_f32 v58, 0, v58\n\t" \
        "v_max_f32 v59, 0, v59\n\t" \
        "v_max_f32 v60, 0, v60\n\t" \
        "v_max_f32 v61, 0, v61\n\t" \
        "v_max_f32 v62, 0, v62\n\t" \
        "v_max_f32 v63, 0, v63\n\t" \
        "v_max_f32 v64, 0, v64\n\t" \
        "v_max_f32 v65, 0, v65\n\t" \
        "v_max_f32 v66, 0, v66\n\t" \
        "v_max_f32 v67, 0, v67\n\t" \
        "v_max_f32 v68, 0, v68\n\t" \
        "v_max_f32 v69, 0, v69\n\t" \
        "v_max_f32 v70, 0, v70\n\t" \
        "v_max_f32 v71, 0, v71\n\t" \
        "v_max_f32 v72, 0, v72\n\t" \
        "v_max_f32 v73, 0, v73\n\t" \
        "v_max_f32 v74, 0, v74\n\t" \
        "v_max_f32 v75, 0, v75\n\t" \
        "v_max_f32 v76, 0, v76\n\t" \
        "v_max_f32 v77, 0, v77\n\t" \
        "v_max_f32 v78, 0, v78\n\t" \
        "v_max_f32 v79, 0, v79\n\t" \
        "v_max_f32 v80, 0, v80\n\t" \
        "v_max_f32 v81, 0, v81\n\t" \
        "v_max_f32 v82, 0, v82\n\t" \
        "v_max_f32 v83, 0, v83\n\t" \
        "v_max_f32 v84, 0, v84\n\t" \
        "v_max_f32 v85, 0, v85\n\t" \
        "v_max_f32 v86, 0, v86\n\t" \
        "v_max_f32 v87, 0, v87\n\t" \
        "v_max_f32 v88, 0, v88\n\t" \
        "v_max_f32 v89, 0, v89\n\t" \
        "v_max_f32 v90, 0, v90\n\t" \
        "v_max_f32 v91, 0, v91\n\t" \
        "v_max_f32 v92, 0, v92\n\t" \
        "v_max_f32 v93, 0, v93\n\t" \
        "v_max_f32 v94, 0, v94\n\t" \
        "v_max_f32 v95, 0, v95\n\t" \
        "v_max_f32 v96, 0, v96\n\t" \
        "v_max_f32 v97, 0, v97\n\t" \
        "v_max_f32 v98, 0, v98\n\t" \
        "v_max_f32 v99, 0, v99\n\t" \
        "v_max_f32 v100, 0, v100\n\t" \
        "v_max_f32 v101, 0, v101\n\t" \
        "v_max_f32 v102, 0, v102\n\t" \
        "v_max_f32 v103, 0, v103\n\t" \
        "v_max_f32 v104, 0, v104\n\t" \
        "v_max_f32 v105, 0, v105\n\t" \
        "v_max_f32 v106, 0, v106\n\t" \
        "v_max_f32 v107, 0, v107\n\t" \
        "v_max_f32 v108, 0, v108\n\t" \
        "v_max_f32 v109, 0, v109\n\t" \
        "v_max_f32 v110, 0, v110\n\t" \
        "v_max_f32 v111, 0, v111\n\t" \
        "v_max_f32 v112, 0, v112\n\t" \
        "v_max_f32 v113, 0, v113\n\t" \
        "v_max_f32 v114, 0, v114\n\t" \
        "v_max_f32 v115, 0, v115\n\t" \
        "v_max_f32 v116, 0, v116\n\t" \
        "v_max_f32 v117, 0, v117\n\t" \
        "v_max_f32 v118, 0, v118\n\t" \
        "v_max_f32 v119, 0, v119\n\t" \
        "v_max_f32 v120, 0, v120\n\t" \
        "v_max_f32 v121, 0, v121\n\t" \
        "v_max_f32 v122, 0, v122\n\t" \
        "v_max_f32 v123, 0, v123\n\t" \
        "v_max_f32 v124, 0, v124\n\t" \
        "v_max_f32 v125, 0, v125\n\t" \
        "v_max_f32 v126, 0, v126\n\t" \
        "v_max_f32 v127, 0, v127\n\t" \
        "v_max_f32 v128, 0, v128\n\t" \
        "v_max_f32 v129, 0, v129\n\t" \
        "v_max_f32 v130, 0, v130\n\t" \
        "v_max_f32 v131, 0, v131\n\t" \
        "v_max_f32 v132, 0, v132\n\t" \
        "v_max_f32 v133, 0, v133\n\t" \
        "v_max_f32 v134, 0, v134\n\t" \
        "v_max_f32 v135, 0, v135\n\t" \
        "v_max_f32 v136, 0, v136\n\t" \
        "v_max_f32 v137, 0, v137\n\t" \
        "v_max_f32 v138, 0, v138\n\t" \
        "v_max_f32 v139, 0, v139\n\t" \
        "v_max_f32 v140, 0, v140\n\t" \
        "v_max_f32 v141, 0, v141\n\t" \
        "v_max_f32 v142, 0, v142\n\t" \
        "v_max_f32 v143, 0, v143\n\t" \
        "v_max_f32 v144, 0, v144\n\t" \
        "v_max_f32 v145, 0, v145\n\t" \
        "v_max_f32 v146, 0, v146\n\t" \
        "v_max_f32 v147, 0, v147\n\t" \
        "v_max_f32 v148, 0, v148\n\t" \
        "v_max_f32 v149, 0, v149\n\t" \
        "v_max_f32 v150, 0, v150\n\t" \
        "v_max_f32 v151, 0, v151\n\t" \
        "v_max_f32 v152, 0, v152\n\t" \
        "v_max_f32 v153, 0, v153\n\t" \
        "v_max_f32 v154, 0, v154\n\t" \
        "v_max_f32 v155, 0, v155\n\t" \
        "v_max_f32 v156, 0, v156\n\t" \
        "v_max_f32 v157, 0, v157\n\t" \
        "v_max_f32 v158, 0, v158\n\t" \
        "v_max_f32 v159, 0, v159\n\t"
#define QUANT \
        "s_mov_b64 s[40:41], %[ob]\n\t" \
        "s_mov_b32 s42, 0x0c0c0400\n\t" \
        "s_mov_b32 s43, 0x04000c0c\n\t" \
        "s_waitcnt vmcnt(0)\n\t" \
        "v_pk_fma_f32 v[176:177], v[0:1], %[iv], %[mg]\n\t" \
        "v_pk_fma_f32 v[178:179], v[2:3], %[iv], %[mg]\n\t" \
        "v_pk_fma_f32 v[180:181], v[4:5], %[iv], %[mg]\n\t" \
        "v_pk_fma_f32 v[182:183], v[6:7], %[iv], %[mg]\n\t" \
        "v_pk_fma_f32 v[184:185], v[8:9], %[iv], %[mg]\n\t" \
        "v_pk_fma_f32 v[186:187], v[10:11], %[iv], %[mg]\n\t" \
        "v_pk_fma_f32 v[188:189], v[12:13], %[iv], %[mg]\n\t" \
        "v_pk_fma_f32 v[190:191], v[14:15], %[iv], %[mg]\n\t" \
        "v_perm_b32 v176, v180, v176, s42\n\t" \
        "v_perm_b32 v184, v188, v184, s43\n\t" \
        "v_perm_b32 v177, v181, v177, s42\n\t" \
        "v_perm_b32 v185, v189, v185, s43\n\t" \
        "v_perm_b32 v178, v182, v178, s42\n\t" \
        "v_perm_b32 v186, v190, v186, s43\n\t" \
        "v_perm_b32 v179, v183, v179, s42\n\t" \
        "v_perm_b32 v187, v191, v187, s43\n\t" \
        "v_or_b32 v160, v176, v184\n\t" \
        "v_or_b32 v161, v177, v185\n\t" \
        "v_or_b32 v162, v178, v186\n\t" \
        "v_or_b32 v163, v179, v187\n\t" \
        "ds_write_b128 %[lw], v[160:163] offset:0\n\t" \
        "v_pk_fma_f32 v[176:177], v[16:17], %[iv], %[mg]\n\t" \
        "v_pk_fma_f32 v[178:179], v[18:19], %[iv], %[mg]\n\t" \
        "v_pk_fma_f32 v[180:181], v[20:21], %[iv], %[mg]\n\t" \
        "v_pk_fma_f32 v[182:183], v[22:23], %[iv], %[mg]\n\t" \
        "v_pk_fma_f32 v[184:185], v[24:25], %[iv], %[mg]\n\t" \
        "v_pk_fma_f32 v[186:187], v[26:27], %[iv], %[mg]\n\t" \
        "v_pk_fma_f32 v[188:189], v[28:29], %[iv], %[mg]\n\t" \
        "v_pk_fma_f32 v[190:191], v[30:31], %[iv], %[mg]\n\t" \
        "v_perm_b32 v176, v180, v176, s42\n\t" \
        "v_perm_b32 v184, v188, v184, s43\n\t" \
        "v_perm_b32 v177, v181, v177, s42\n\t" \
        "v_perm_b32 v185, v189, v185, s43\n\t" \
        "v_perm_b32 v178, v182, v178, s42\n\t" \
        "v_perm_b32 v186, v190, v186, s43\n\t" \
        "v_perm_b32 v179, v183, v179, s42\n\t" \
        "v_perm_b32 v187, v191, v187, s43\n\t" \
        "v_or_b32 v164, v176, v184\n\t" \
        "v_or_b32 v165, v177, v185\n\t" \
        "v_or_b32 v166, v178, v186\n\t" \
        "v_or_b32 v167, v179, v187\n\t" \
        "ds_write_b128 %[lw], v[164:167] offset:64\n\t" \
        "ds_read_b128 v[192:195], %[lr] offset:0\n\t" \
        "ds_read_b128 v[196:199], %[lr] offset:1408\n\t" \
        "v_pk_fma_f32 v[176:177], v[128:129], %[iv], %[mg]\n\t" \
        "v_pk_fma_f32 v[178:179], v[130:131], %[iv], %[mg]\n\t" \
        "v_pk_fma_f32 v[180:181], v[132:133], %[iv], %[mg]\n\t" \
        "v_pk_fma_f32 v[182:183], v[134:135], %[iv], %[mg]\n\t" \
        "v_perm_b32 v176, v180, v176, s42\n\t" \
        "v_perm_b32 v177, v181, v177, s43\n\t" \
        "v_perm_b32 v178, v182, v178, s42\n\t" \
        "v_perm_b32 v179, v183, v179, s43\n\t" \
        "v_or_b32 v168, v176, v177\n\t" \
        "v_or_b32 v169, v178, v179\n\t" \
        "ds_write_b64 %[lw8], v[168:169] offset:128\n\t" \
        "v_pk_fma_f32 v[176:177], v[32:33], %[iv], %[mg]\n\t" \
        "v_pk_fma_f32 v[178:179], v[34:35], %[iv], %[mg]\n\t" \
        "v_pk_fma_f32 v[180:181], v[36:37], %[iv], %[mg]\n\t" \
        "v_pk_fma_f32 v[182:183], v[38:39], %[iv], %[mg]\n\t" \
        "v_pk_fma_f32 v[184:185], v[40:41], %[iv], %[mg]\n\t" \
        "v_pk_fma_f32 v[186:187], v[42:43], %[iv], %[mg]\n\t" \
        "v_pk_fma_f32 v[188:189], v[44:45], %[iv], %[mg]\n\t" \
        "v_pk_fma_f32 v[190:191], v[46:47], %[iv], %[mg]\n\t" \
        "v_perm_b32 v176, v180, v176, s42\n\t" \
        "v_perm_b32 v184, v188, v184, s43\n\t" \
        "v_perm_b32 v177, v181, v177, s42\n\t" \
        "v_perm_b32 v185, v189, v185, s43\n\t" \
        "v_perm_b32 v178, v182, v178, s42\n\t" \
        "v_perm_b32 v186, v190, v186, s43\n\t" \
        "v_perm_b32 v179, v183, v179, s42\n\t" \
        "v_perm_b32 v187, v191, v187, s43\n\t" \
        "v_or_b32 v172, v176, v184\n\t" \
        "v_or_b32 v173, v177, v185\n\t" \
        "v_or_b32 v174, v178, v186\n\t" \
        "v_or_b32 v175, v179, v187\n\t" \
        "s_waitcnt lgkmcnt(1)\n\t" \
        "global_store_dwordx4 %[o16], v[192:195], s[40:41] sc0 sc1\n\t" \
        "s_add_u32 s40, s40, 16384\n\t" \
        "s_addc_u32 s41, s41, 0\n\t" \
        "global_store_dwordx4 %[o16], v[196:199], s[40:41] sc0 sc1\n\t" \
        "s_add_u32 s40, s40, 16384\n\t" \
        "s_addc_u32 s41, s41, 0\n\t" \
        "ds_write_b128 %[lw], v[172:175] offset:2816\n\t" \
        "v_pk_fma_f32 v[176:177], v[48:49], %[iv], %[mg]\n\t" \
        "v_pk_fma_f32 v[178:179], v[50:51], %[iv], %[mg]\n\t" \
        "v_pk_fma_f32 v[180:181], v[52:53], %[iv], %[mg]\n\t" \
        "v_pk_fma_f32 v[182:183], v[54:55], %[iv], %[mg]\n\t" \
        "v_pk_fma_f32 v[184:185], v[56:57], %[iv], %[mg]\n\t" \
        "v_pk_fma_f32 v[186:187], v[58:59], %[iv], %[mg]\n\t" \
        "v_pk_fma_f32 v[188:189], v[60:61], %[iv], %[mg]\n\t" \
        "v_pk_fma_f32 v[190:191], v[62:63], %[iv], %[mg]\n\t" \
        "v_perm_b32 v176, v180, v176, s42\n\t" \
        "v_perm_b32 v184, v188, v184, s43\n\t" \
        "v_perm_b32 v177, v181, v177, s42\n\t" \
        "v_perm_b32 v185, v189, v185, s43\n\t" \
        "v_perm_b32 v178, v182, v178, s42\n\t" \
        "v_perm_b32 v186, v190, v186, s43\n\t" \
        "v_perm_b32 v179, v183, v179, s42\n\t" \
        "v_perm_b32 v187, v191, v187, s43\n\t" \
        "v_or_b32 v160, v176, v184\n\t" \
        "v_or_b32 v161, v177, v185\n\t" \
        "v_or_b32 v162, v178, v186\n\t" \
        "v_or_b32 v163, v179, v187\n\t" \
        "ds_write_b128 %[lw], v[160:163] offset:2880\n\t" \
        "ds_read_b128 v[200:203], %[lr] offset:2816\n\t" \
        "ds_read_b128 v[204:207], %[lr] offset:4224\n\t" \
        "v_pk_fma_f32 v[176:177], v[136:137], %[iv], %[mg]\n\t" \
        "v_pk_fma_f32 v[178:179], v[138:139], %[iv], %[mg]\n\t" \
        "v_pk_fma_f32 v[180:181], v[140:141], %[iv], %[mg]\n\t" \
        "v_pk_fma_f32 v[182:183], v[142:143], %[iv], %[mg]\n\t" \
        "v_perm_b32 v176, v180, v176, s42\n\t" \
        "v_perm_b32 v177, v181, v177, s43\n\t" \
        "v_perm_b32 v178, v182, v178, s42\n\t" \
        "v_perm_b32 v179, v183, v179, s43\n\t" \
        "v_or_b32 v164, v176, v177\n\t" \
        "v_or_b32 v165, v178, v179\n\t" \
        "ds_write_b64 %[lw8], v[164:165] offset:2944\n\t" \
        "v_pk_fma_f32 v[176:177], v[64:65], %[iv], %[mg]\n\t" \
        "v_pk_fma_f32 v[178:179], v[66:67], %[iv], %[mg]\n\t" \
        "v_pk_fma_f32 v[180:181], v[68:69], %[iv], %[mg]\n\t" \
        "v_pk_fma_f32 v[182:183], v[70:71], %[iv], %[mg]\n\t" \
        "v_pk_fma_f32 v[184:185], v[72:73], %[iv], %[mg]\n\t" \
        "v_pk_fma_f32 v[186:187], v[74:75], %[iv], %[mg]\n\t" \
        "v_pk_fma_f32 v[188:189], v[76:77], %[iv], %[mg]\n\t" \
        "v_pk_fma_f32 v[190:191], v[78:79], %[iv], %[mg]\n\t" \
        "v_perm_b32 v176, v180, v176, s42\n\t" \
        "v_perm_b32 v184, v188, v184, s43\n\t" \
        "v_perm_b32 v177, v181, v177, s42\n\t" \
        "v_perm_b32 v185, v189, v185, s43\n\t" \
        "v_perm_b32 v178, v182, v178, s42\n\t" \
        "v_perm_b32 v186, v190, v186, s43\n\t" \
        "v_perm_b32 v179, v183, v179, s42\n\t" \
        "v_perm_b32 v187, v191, v187, s43\n\t" \
        "v_or_b32 v168, v176, v184\n\t" \
        "v_or_b32 v169, v177, v185\n\t" \
        "v_or_b32 v170, v178, v186\n\t" \
        "v_or_b32 v171, v179, v187\n\t" \
        "s_waitcnt lgkmcnt(1)\n\t" \
        "global_store_dwordx4 %[o16], v[200:203], s[40:41] sc0 sc1\n\t" \
        "s_add_u32 s40, s40, 16384\n\t" \
        "s_addc_u32 s41, s41, 0\n\t" \
        "global_store_dwordx4 %[o16], v[204:207], s[40:41] sc0 sc1\n\t" \
        "s_add_u32 s40, s40, 16384\n\t" \
        "s_addc_u32 s41, s41, 0\n\t" \
        "ds_write_b128 %[lw], v[168:171] offset:5632\n\t" \
        "v_pk_fma_f32 v[176:177], v[80:81], %[iv], %[mg]\n\t" \
        "v_pk_fma_f32 v[178:179], v[82:83], %[iv], %[mg]\n\t" \
        "v_pk_fma_f32 v[180:181], v[84:85], %[iv], %[mg]\n\t" \
        "v_pk_fma_f32 v[182:183], v[86:87], %[iv], %[mg]\n\t" \
        "v_pk_fma_f32 v[184:185], v[88:89], %[iv], %[mg]\n\t" \
        "v_pk_fma_f32 v[186:187], v[90:91], %[iv], %[mg]\n\t" \
        "v_pk_fma_f32 v[188:189], v[92:93], %[iv], %[mg]\n\t" \
        "v_pk_fma_f32 v[190:191], v[94:95], %[iv], %[mg]\n\t" \
        "v_perm_b32 v176, v180, v176, s42\n\t" \
        "v_perm_b32 v184, v188, v184, s43\n\t" \
        "v_perm_b32 v177, v181, v177, s42\n\t" \
        "v_perm_b32 v185, v189, v185, s43\n\t" \
        "v_perm_b32 v178, v182, v178, s42\n\t" \
        "v_perm_b32 v186, v190, v186, s43\n\t" \
        "v_perm_b32 v179, v183, v179, s42\n\t" \
        "v_perm_b32 v187, v191, v187, s43\n\t" \
        "v_or_b32 v172, v176, v184\n\t" \
        "v_or_b32 v173, v177, v185\n\t" \
        "v_or_b32 v174, v178, v186\n\t" \
        "v_or_b32 v175, v179, v187\n\t" \
        "ds_write_b128 %[lw], v[172:175] offset:5696\n\t" \
        "ds_read_b128 v[208:211], %[lr] offset:5632\n\t" \
        "ds_read_b128 v[212:215], %[lr] offset:7040\n\t" \
        "v_pk_fma_f32 v[176:177], v[144:145], %[iv], %[mg]\n\t" \
        "v_pk_fma_f32 v[178:179], v[146:147], %[iv], %[mg]\n\t" \
        "v_pk_fma_f32 v[180:181], v[148:149], %[iv], %[mg]\n\t" \
        "v_pk_fma_f32 v[182:183], v[150:151], %[iv], %[mg]\n\t" \
        "v_perm_b32 v176, v180, v176, s42\n\t" \
        "v_perm_b32 v177, v181, v177, s43\n\t" \
        "v_perm_b32 v178, v182, v178, s42\n\t" \
        "v_perm_b32 v179, v183, v179, s43\n\t" \
        "v_or_b32 v160, v176, v177\n\t" \
        "v_or_b32 v161, v178, v179\n\t" \
        "ds_write_b64 %[lw8], v[160:161] offset:5760\n\t" \
        "v_pk_fma_f32 v[176:177], v[96:97], %[iv], %[mg]\n\t" \
        "v_pk_fma_f32 v[178:179], v[98:99], %[iv], %[mg]\n\t" \
        "v_pk_fma_f32 v[180:181], v[100:101], %[iv], %[mg]\n\t" \
        "v_pk_fma_f32 v[182:183], v[102:103], %[iv], %[mg]\n\t" \
        "v_pk_fma_f32 v[184:185], v[104:105], %[iv], %[mg]\n\t" \
        "v_pk_fma_f32 v[186:187], v[106:107], %[iv], %[mg]\n\t" \
        "v_pk_fma_f32 v[188:189], v[108:109], %[iv], %[mg]\n\t" \
        "v_pk_fma_f32 v[190:191], v[110:111], %[iv], %[mg]\n\t" \
        "v_perm_b32 v176, v180, v176, s42\n\t" \
        "v_perm_b32 v184, v188, v184, s43\n\t" \
        "v_perm_b32 v177, v181, v177, s42\n\t" \
        "v_perm_b32 v185, v189, v185, s43\n\t" \
        "v_perm_b32 v178, v182, v178, s42\n\t" \
        "v_perm_b32 v186, v190, v186, s43\n\t" \
        "v_perm_b32 v179, v183, v179, s42\n\t" \
        "v_perm_b32 v187, v191, v187, s43\n\t" \
        "v_or_b32 v164, v176, v184\n\t" \
        "v_or_b32 v165, v177, v185\n\t" \
        "v_or_b32 v166, v178, v186\n\t" \
        "v_or_b32 v167, v179, v187\n\t" \
        "s_waitcnt lgkmcnt(1)\n\t" \
        "global_store_dwordx4 %[o16], v[208:211], s[40:41] sc0 sc1\n\t" \
        "s_add_u32 s40, s40, 16384\n\t" \
        "s_addc_u32 s41, s41, 0\n\t" \
        "global_store_dwordx4 %[o16], v[212:215], s[40:41] sc0 sc1\n\t" \
        "s_add_u32 s40, s40, 16384\n\t" \
        "s_addc_u32 s41, s41, 0\n\t" \
        "ds_write_b128 %[lw], v[164:167] offset:8448\n\t" \
        "v_pk_fma_f32 v[176:177], v[112:113], %[iv], %[mg]\n\t" \
        "v_pk_fma_f32 v[178:179], v[114:115], %[iv], %[mg]\n\t" \
        "v_pk_fma_f32 v[180:181], v[116:117], %[iv], %[mg]\n\t" \
        "v_pk_fma_f32 v[182:183], v[118:119], %[iv], %[mg]\n\t" \
        "v_pk_fma_f32 v[184:185], v[120:121], %[iv], %[mg]\n\t" \
        "v_pk_fma_f32 v[186:187], v[122:123], %[iv], %[mg]\n\t" \
        "v_pk_fma_f32 v[188:189], v[124:125], %[iv], %[mg]\n\t" \
        "v_pk_fma_f32 v[190:191], v[126:127], %[iv], %[mg]\n\t" \
        "v_perm_b32 v176, v180, v176, s42\n\t" \
        "v_perm_b32 v184, v188, v184, s43\n\t" \
        "v_perm_b32 v177, v181, v177, s42\n\t" \
        "v_perm_b32 v185, v189, v185, s43\n\t" \
        "v_perm_b32 v178, v182, v178, s42\n\t" \
        "v_perm_b32 v186, v190, v186, s43\n\t" \
        "v_perm_b32 v179, v183, v179, s42\n\t" \
        "v_perm_b32 v187, v191, v187, s43\n\t" \
        "v_or_b32 v168, v176, v184\n\t" \
        "v_or_b32 v169, v177, v185\n\t" \
        "v_or_b32 v170, v178, v186\n\t" \
        "v_or_b32 v171, v179, v187\n\t" \
        "ds_write_b128 %[lw], v[168:171] offset:8512\n\t" \
        "ds_read_b128 v[216:219], %[lr] offset:8448\n\t" \
        "ds_read_b128 v[220:223], %[lr] offset:9856\n\t" \
        "v_pk_fma_f32 v[176:177], v[152:153], %[iv], %[mg]\n\t" \
        "v_pk_fma_f32 v[178:179], v[154:155], %[iv], %[mg]\n\t" \
        "v_pk_fma_f32 v[180:181], v[156:157], %[iv], %[mg]\n\t" \
        "v_pk_fma_f32 v[182:183], v[158:159], %[iv], %[mg]\n\t" \
        "v_perm_b32 v176, v180, v176, s42\n\t" \
        "v_perm_b32 v177, v181, v177, s43\n\t" \
        "v_perm_b32 v178, v182, v178, s42\n\t" \
        "v_perm_b32 v179, v183, v179, s43\n\t" \
        "v_or_b32 v172, v176, v177\n\t" \
        "v_or_b32 v173, v178, v179\n\t" \
        "ds_write_b64 %[lw8], v[172:173] offset:8576\n\t" \
        "ds_read_b128 v[224:227], %[lr8] offset:128\n\t" \
        "ds_read_b128 v[228:231], %[lr8] offset:5760\n\t" \
        "s_waitcnt lgkmcnt(3)\n\t" \
        "global_store_dwordx4 %[o16], v[216:219], s[40:41] sc0 sc1\n\t" \
        "s_add_u32 s40, s40, 16384\n\t" \
        "s_addc_u32 s41, s41, 0\n\t" \
        "global_store_dwordx4 %[o16], v[220:223], s[40:41] sc0 sc1\n\t" \
        "s_add_u32 s40, s40, 16384\n\t" \
        "s_addc_u32 s41, s41, 0\n\t" \
        "s_cmp_eq_u32 %[nt], 12\n\t" \
        "s_waitcnt lgkmcnt(0)\n\t" \
        "s_cbranch_scc1 LQ%=\n\t" \
        "s_mov_b64 s[40:41], %[ob]\n\t" \
        "global_store_dwordx4 %[o8], v[224:227], s[40:41]\n\t" \
        "s_add_u32 s40, s40, 65536\n\t" \
        "s_addc_u32 s41, s41, 0\n\t" \
        "global_store_dwordx4 %[o8], v[228:231], s[40:41]\n\t" \
        "\nLQ%=:\n\t" \
        "s_nop 1\n\t"
#define ACC_RW "+{v[0:15]}"(acc[0]), "+{v[16:31]}"(acc[1]), "+{v[32:47]}"(acc[2]), "+{v[48:63]}"(acc[3]), "+{v[64:79]}"(acc[4]), \
               "+{v[80:95]}"(acc[5]), "+{v[96:111]}"(acc[6]), "+{v[112:127]}"(acc[7]), "+{v[128:143]}"(acc[8]), "+{v[144:159]}"(acc[9])

__global__ void __launch_bounds__(512, 2) ag_gemm_kernel(Params p) {
    __shared__ __attribute__((aligned(16))) unsigned char smem[L_TOTAL];
    const unsigned lbase = (unsigned)(uintptr_t)(LAS unsigned char*)smem;
    const int lane = threadIdx.x & 63, wave = rfl(threadIdx.x >> 6);
    const int b = blockIdx.x, xcd = b & 7, j8 = b >> 3;
    asm volatile("" :: "s"(p.x), "s"(p.w), "s"(p.own), "s"(p.rank));
    const int rank = p.rank;
    char* const ws = p.own;
    unsigned* const tmo = (unsigned*)(ws + WS_TMO);
    const bool gemm = b < NGEMM;
    const int midx = (xcd >> 1) * 4 + (j8 < 24 ? j8 & 3 : (xcd & 1) * 2 + (j8 & 1)), nt = j8 < 24 ? (xcd & 1) * 7 + (j8 >> 2) : 6;
    const int mt = (rank * 2 + midx) & 15;
    const u64 tstart = rt();
    unsigned* const cnt = (unsigned*)(ws + WS_CNT) + b * 8 + wave;
    const unsigned callm1 = __hip_atomic_load(cnt, RLX_AGENT);
    unsigned spins = 0;

    if (!gemm) {
        const int pc = b - NGEMM - NPULL;
        if (wave >= 6) pack_x<1>(p, ws, pc < 0 ? NPREP + b - NGEMM : pc, 7 - wave, lane, cnt, (LAS unsigned char*)smem + L_PACK);
        else {
            if (pc < 0) for (int i = 0; i < 4; ++i) ((LAS u32x4*)smem)[wave * 256 + i * 64 + lane] = (u32x4){0u, 0u, 0u, 0u};
            asm volatile("s_waitcnt lgkmcnt(0)\n\ts_barrier\n\ts_barrier" ::: "memory");
        }
    } else {
        ((LAS unsigned*)smem)[L_CTRL / 4 + threadIdx.x] = threadIdx.x >= 256 ? (unsigned)RING : 0u;
        asm volatile("s_waitcnt lgkmcnt(0)\n\ts_barrier" ::: "memory");
        if (wave == 4) {
            constexpr int U1 = NPREP + NPULL;
            __builtin_amdgcn_s_setprio(3);
            while (rt() < tstart + XDL) __builtin_amdgcn_s_sleep(1);
            for (int i = 0; i < 5; ++i) {
                int u = NUNIT;
                if (i == 0) u = U1 + b;
                else if (i < 3) { if (!(b & 1)) u = U1 + NGEMM + b + i - 1; }
                else if (midx < 2) u = U1 + 2 * NGEMM + (i - 3) * 2 * NT + midx * NT + nt;
                if (u < NUNIT) pack_x<2>(p, ws, u, 0, lane, cnt, (LAS unsigned char*)smem + L_PACK);
                __builtin_amdgcn_s_setprio(0);
            }
        } else if (wave < 2 || wave == 6) {
            if (wave < 2) __builtin_amdgcn_s_setprio(1);
            if (wave) while (rt() < tstart + (wave == 1 ? WDL1 : WDL)) __builtin_amdgcn_s_sleep(2);
            for (int wg = wave < 2 ? wave : 2; wg < (wave < 2 ? wave + 1 : W1); ++wg) conv_w(p.w, ws, nt, wg * 16 + midx, lane, cnt);
        }
    }
    const unsigned call = rfl(callm1) + 1u;
    if (b < NGEMM + NPULL && lane == 0) __hip_atomic_store(cnt, call, RLX_AGENT);

    if (wave < 4 && gemm) {
        __builtin_amdgcn_s_setprio(1);
        f32x16 acc[10];
        unsigned vf0 = 0;
        {
            const unsigned vlw = lbase + lane * 16, vl16 = lane * 16;
            const unsigned ago = lbase + L_GO + (lane & 1) * 4, afree = lbase + L_FREE + lane * 16 + wave * 4;
            const char* xsrc = ws + WS_XB + (size_t)mt * XTS + wave * 4096;
            unsigned vaw, vax, vt, ss, sb;
            asm volatile(MAINLOOP
                : "=&{v[0:15]}"(acc[0]), "=&{v[16:31]}"(acc[1]), "=&{v[32:47]}"(acc[2]), "=&{v[48:63]}"(acc[3]), "=&{v[64:79]}"(acc[4]),
                  "=&{v[80:95]}"(acc[5]), "=&{v[96:111]}"(acc[6]), "=&{v[112:127]}"(acc[7]), "=&{v[128:143]}"(acc[8]), "=&{v[144:159]}"(acc[9]),
                  [vaw] "=&v"(vaw), [vax] "=&v"(vax), [vf0] "=&v"(vf0), [vt] "=&v"(vt), [ss] "=&s"(ss), [sb] "=&s"(sb)
                : [vlw] "v"(vlw), [vl16] "v"(vl16), [ago] "v"(ago), [afree] "v"(afree), [xs] "s"(xsrc)
                : "memory", "scc", "vcc", "s40", "s41", "s42", "s43", "v160", "v161", "v162", "v163", "v164", "v165", "v166", "v167", "v168", "v169", "v170", "v171", "v172", "v173", "v174", "v175", "v176", "v177", "v178", "v179", "v180", "v181", "v182", "v183", "v184", "v185", "v186", "v187", "v188", "v189", "v190", "v191", "v192", "v193", "v194", "v195", "v196", "v197", "v198", "v199", "v200", "v201", "v202", "v203", "v204", "v205", "v206", "v207", "v208", "v209", "v210", "v211", "v212", "v213", "v214", "v215", "v216", "v217", "v218", "v219", "v220", "v221", "v222", "v223", "v224", "v225", "v226", "v227", "v228", "v229", "v230", "v231", "v232", "v233", "v234", "v235", "v236", "v237", "v238", "v239", "v240", "v241", "v242", "v243");
        }
        const unsigned wm = wave_max_u32(vf0);
        if (lane == 0) {
            unsigned old, tm;
            asm volatile("ds_max_u32 %2, %3\n\tds_add_rtn_u32 %0, %2, %4 offset:4\n\tds_read_b32 %1, %2\n\ts_waitcnt lgkmcnt(0)"
                         : "=&v"(old), "=&v"(tm) : "v"(lbase + L_WMAX), "v"(wm), "v"(1u) : "memory");
            if (old == 3u) __hip_atomic_store((u64*)(ws + WS_AML) + (call & 1) * NWG + b, ((u64)call << 32) | tm, RLX_SYS);
        }
        asm volatile(RELU : ACC_RW);
        signed char* const ob = p.out + (size_t)(mt * 256 + wave * 64) * NLOC + nt * 128;
        const unsigned lw = lbase + wave * 12288 + (lane & 15) * 176 + (lane >> 4) * 16, lw8 = lw - (lane >> 4) * 8;
        const unsigned lr = lbase + wave * 12288 + (lane >> 3) * 176 + (lane & 7) * 16, lr8 = lbase + wave * 12288 + (lane >> 1) * 176 + (lane & 1) * 16;
        const unsigned o16 = (lane >> 3) * NLOC + (lane & 7) * 16, o8 = (lane >> 1) * NLOC + (lane & 1) * 16 + 1664 - 96 * nt;
        typedef float f32x2 __attribute__((ext_vector_type(2)));
        const f32x2 magic = {12582912.f, 12582912.f};
        unsigned used = ~0u, state;
        do {
            u32x2 fin;
            asm volatile("ds_read_b64 %0, %1\n\ts_waitcnt lgkmcnt(0)" : "=v"(fin) : "v"(lbase + L_FIN), "v"(o16), "v"(o8), "v"(magic), "s"(ob), "v"(lw), "v"(lw8), "v"(lr), "v"(lr8) : "memory");
            state = rfl(fin.y);
            if ((state & 1u) && rfl(fin.x) != used) {
                used = rfl(fin.x);
                const u32x2 iv = {fin.x, fin.x};
                asm volatile(QUANT : ACC_RW : [iv] "v"(iv), [mg] "v"(magic), [o16] "v"(o16), [o8] "v"(o8), [ob] "s"(ob), [nt] "s"(nt),
                               [lw] "v"(lw), [lw8] "v"(lw8), [lr] "v"(lr), [lr8] "v"(lr8)
                             : "memory", "scc", "s40", "s41", "s42", "s43", "v160", "v161", "v162", "v163", "v164", "v165", "v166", "v167", "v168", "v169", "v170", "v171", "v172", "v173", "v174", "v175", "v176", "v177", "v178", "v179", "v180", "v181", "v182", "v183", "v184", "v185", "v186", "v187", "v188", "v189", "v190", "v191", "v192", "v193", "v194", "v195", "v196", "v197", "v198", "v199", "v200", "v201", "v202", "v203", "v204", "v205", "v206", "v207", "v208", "v209", "v210", "v211", "v212", "v213", "v214", "v215", "v216", "v217", "v218", "v219", "v220", "v221", "v222", "v223", "v224", "v225", "v226", "v227", "v228", "v229", "v230", "v231");
            }
        } while (state == 0u || state == 3u);
        if (state == 2u) {
            VM0();
            const u32x4 mark = {0x80808080u, 0x80808080u, 0x80808080u, 0x80808080u};
            for (int j = 0; j < 8; ++j) *(u32x4*)(ob + (size_t)j * 8 * NLOC + o16) = mark;
            if (nt < NT - 1) for (int j = 0; j < 2; ++j) *(u32x4*)(ob + (size_t)j * 32 * NLOC + o8) = mark;
        }
    } else if (b >= NGEMM + NPULL) {
        constexpr unsigned NJ = (NST - 16 * W1) * NT;
        for (unsigned u = (b - NGEMM - NPULL) * 8 + wave; u < NJ;) {
            unsigned t = 0;
            if (lane == 0) t = __hip_atomic_fetch_add((unsigned*)(ws + WS_TK + rank * 128), 1u, RLX_AGENT);
            const unsigned tn = (unsigned)(rt() - tstart);
            if (tn > PP0 && tn < PP1 && u > NPREP * 8) {
                const unsigned need = (NJ - u + NPREP * 4) * (tn - 400u) / (u - NPREP * 4);
                while ((unsigned)(rt() - tstart) < min((unsigned)PP1, TEND - min(need, (unsigned)TEND))) __builtin_amdgcn_s_sleep(16);
            }
            conv_w(p.w, ws, u % NT, 16 * W1 + u / NT, lane, cnt);
            u = rfl(t) - (call - 1u) * NJ + NPREP * 8;
        }
        if (lane == 0) __hip_atomic_store(cnt, call, RLX_AGENT);
        if (b >= NWG - 2 && wave == 7) {
            while (rt() < tstart + 4000) __builtin_amdgcn_s_sleep(32);
            const u64* gp = (const u64*)(ws + WS_AML) + (call & 1) * NWG + min(lane * 4, NGEMM - 4);
            const bool early = b == NWG - 2;
            GAS u64* const dst = (GAS u64*)(lane_peer(p, lane) + WS_AMG + (((call & 1) * 2 + early) * NR + rank) * 128);
            const u64 t0 = rt();
            unsigned sent = ~0u;
            for (bool done = false; !done;) {
                u32x4 e0, e1;
                asm volatile("global_load_dwordx4 %0, %2, off sc0 sc1\n\tglobal_load_dwordx4 %1, %2, off offset:16 sc0 sc1\n\ts_waitcnt vmcnt(0)" : "=&v"(e0), "=&v"(e1) : "v"(gp) : "memory");
                done = __all(e0.y == call && e0.w == call && e1.y == call && e1.w == call) || give_up(tmo, call, t0, spins);
                if (early ? !done && __builtin_popcountll(__ballot(e0.y == call) & ((1ull << 52) - 1)) + __builtin_popcountll(__ballot(e1.w == call) & ((1ull << 52) - 1)) >= SPEC : done) {
                    const unsigned m = wave_max_u32(max(max(e0.y == call ? e0.x : 0u, e0.w == call ? e0.z : 0u), max(e1.y == call ? e1.x : 0u, e1.w == call ? e1.z : 0u)));
                    if (m != sent && lane < NR) __hip_atomic_store(dst, ((u64)call << 32) | m, RLX_SYS);
                    sent = m;
                }
            }
        }
    } else if (!gemm) {
        if (wave == 7 && b - NGEMM < 8 && (b - NGEMM) * 64 >= NDIRECT) {
            const size_t off = WS_HDR + (size_t)(rank * NUNIT + (b - NGEMM) * 64 + lane) * 8;
            const u64 t0 = rt();
            for (;;) {
                const u64 h = __hip_atomic_load((const u64*)(ws + off), RLX_SYS);
                if (__all((unsigned)(h >> 32) == call)) {
#pragma unroll
                    for (int i = 1; i < NR; ++i) __hip_atomic_store((u64*)(p.ws[(rank + i) & 7] + off), h, RLX_SYS);
                    break;
                }
                if (give_up(tmo, call, t0, spins)) break;
                __builtin_amdgcn_s_sleep(16);
            }
        }
        const int v = wave * NPULL + b - NGEMM;
        if (wave < 7 && v < 7 * KP) {
            const int g = v / 7, srcr = (rank + 1 + v % 7) & 7;
            const char* const rpk = p.ws[srcr] + WS_PK;
            const char* const rov = p.ws[srcr] + WS_OV;
            const char* const hp = ws + WS_HDR + (size_t)srcr * NUNIT * 8;
            const char* const xb0 = ws + WS_XB + (size_t)srcr * 2 * XTS;
            const char* const fx0 = ws + WS_FX + (size_t)srcr * 2 * NST * 8;
            const char* const tk = ws + WS_TK + srcr * 128;
            const char* const dum = ws + WS_DUM + v * 128;
            const unsigned tbase = (call - 1u) * (unsigned)(NUNIT - NPUSH - 2 * KP) - (NPUSH + 3u * KP);
            unsigned cur = NPUSH + g, nxt = cur + KP, nn = nxt + KP;
            while (cur < (unsigned)NUNIT) {
                u32x2 h;
                const u64 t0 = rt();
                for (;;) {
                    asm volatile("s_load_dwordx2 %0, %1, 0x0 glc\n\ts_waitcnt lgkmcnt(0)" : "=&s"(h) : "s"(hp + (size_t)cur * 8) : "memory");
                    if (h.y == call) break;
                    if (give_up(tmo, call, t0, spins)) return;
                }
                cur = rfl(cur); nxt = rfl(nxt); nn = rfl(nn);
                asm volatile(PULL
                             : [cur] "+s"(cur), [nxt] "+s"(nxt), [nn] "+s"(nn)
                             : [meta] "s"(h.x), [call] "s"(call), [tb] "s"(tbase), [pk] "s"(rpk), [ov] "s"(rov), [hp] "s"(hp), [xb] "s"(xb0), [fx] "s"(fx0),
                               [tk] "s"(tk), [dum] "s"(dum), [lane] "v"(lane), [lds] "s"(lbase + wave * 4096), [sk] "n"(XSKEW)
                             : "memory", "scc", "vcc", "v8", "v9", "v10", "v11", "v12", "v13", "v14", "v15", "v16", "v17", "v18", "v19", "v20", "v21", "v22", "v23", "v24", "v25", "v26", "v27", "v28", "v29", "v30", "v31", "v32", "v33", "v34", "v35", "v36", "v37", "v38", "v39", "v40", "v41", "v42", "v43", "v44", "v45", "v46", "v47", "v48", "v49", "v50", "v51", "v52", "v53", "v54", "v55", "v56", "v57", "v58", "v59", "v60", "v61", "v62", "v63", "v64", "v65", "v66", "v67", "v68", "v69", "v70", "v71", "v72", "v73", "v74", "v75", "v76", "v77", "v78", "v79", "v80", "v81", "v82", "v83", "v84", "v85", "v86", "v87", "v88", "v89", "v90", "v91", "v92", "v93", "v94", "v95", "v96", "v97", "v98", "v99", "v100", "v101", "v102", "v103", "v104", "v105", "v106", "v107", "v108", "v109", "v110", "v111", "v112", "v113", "v114", "v115", "v116", "v117", "v118", "v119", "v120", "v121", "v122", "v123", "v124", "v125", "v126", "v127", "v128", "v129", "v130", "v131", "v132", "v133", "v134", "v135", "v136", "v137", "v138", "v139", "v140", "v141", "v142", "v143", "v144", "v145", "v146", "v147", "v148", "v149", "v150", "v151", "v152", "v153", "v154", "v155", "v156", "v157", "v158", "v159", "v160", "v161", "v162", "v163", "v164", "v165", "s44", "s45", "s46", "s47", "s48", "s49", "s50", "s51", "s52", "s53", "s54", "s55", "s56", "s57", "s58", "s59", "s60", "s61", "s62", "s63", "s64", "s65", "s66", "s67", "s68", "s69", "s70", "s71", "s72", "s73", "s74", "s75");
            }
        }
    } else if (wave == 6) {
        if (PZ && nt < 3) {
            while (rt() < tstart + PZ + b * 4) __builtin_amdgcn_s_sleep(16);
            signed char* const oz = p.out + (size_t)(mt * 256 + (lane >> 3)) * NLOC + 1664 + nt * 128 + (lane & 7) * 16;
            for (int j = 0; j < 32; ++j) st16(oz + (size_t)j * 8 * NLOC, (u32x4){0u, 0u, 0u, 0u});
            VM0();
            if (lane == 0) __hip_atomic_store((unsigned*)(ws + WS_PZ) + mt * 4 + nt, call, RLX_SYS);
        }
    } else if (wave == 5) {
        loader(ws + WS_WB + (size_t)nt * NST * ST_B, lbase, lane);
    } else if (wave == 7) {
        unsigned ra = 0, rb = 0;
        const unsigned* fx = (const unsigned*)(ws + WS_FX) + mt * NST * 2;
        const unsigned* fw = (const unsigned*)(ws + WS_FW) + nt * NST;
        const unsigned apub = lbase + (lane == 0 ? L_GO : lane == 1 ? L_RDYB : L_DUMMY + 256 + lane * 4);
        u64 t0 = rt();
        bool dead = false;
        while (ra < (unsigned)NST || rb < (unsigned)NST) {
            const unsigned wl = min((unsigned)NST, ra + WAHEAD);
            unsigned na = ra, nb = rb;
            if (rb < wl) nb = rb & ~15u;
            asm volatile("s_cmp_ge_u32 %[rb], %[wl]\n\ts_cbranch_scc1 LPA%=\n\ts_load_dwordx16 s[80:95], %[q], 0x0 glc\n"
                         "LPA%=:\n\ts_cmp_ge_u32 %[n], 128\n\ts_cbranch_scc1 LPB%=\n\ts_load_dwordx16 s[64:79], %[p], 0x0 glc\n\ts_waitcnt lgkmcnt(0)\n\t"
                         "s_min_u32 s64, s64, s65\n\ts_cmp_ge_u32 s64, %[c]\n\ts_addc_u32 %[n], %[n], 0\n\t"
                         "s_min_u32 s64, s64, s66\n\ts_min_u32 s64, s64, s67\n\ts_cmp_ge_u32 s64, %[c]\n\ts_addc_u32 %[n], %[n], 0\n\t"
                         "s_min_u32 s64, s64, s68\n\ts_min_u32 s64, s64, s69\n\ts_cmp_ge_u32 s64, %[c]\n\ts_addc_u32 %[n], %[n], 0\n\t"
                         "s_min_u32 s64, s64, s70\n\ts_min_u32 s64, s64, s71\n\ts_cmp_ge_u32 s64, %[c]\n\ts_addc_u32 %[n], %[n], 0\n\t"
                         "s_min_u32 s64, s64, s72\n\ts_min_u32 s64, s64, s73\n\ts_cmp_ge_u32 s64, %[c]\n\ts_addc_u32 %[n], %[n], 0\n\t"
                         "s_min_u32 s64, s64, s74\n\ts_min_u32 s64, s64, s75\n\ts_cmp_ge_u32 s64, %[c]\n\ts_addc_u32 %[n], %[n], 0\n\t"
                         "s_min_u32 s64, s64, s76\n\ts_min_u32 s64, s64, s77\n\ts_cmp_ge_u32 s64, %[c]\n\ts_addc_u32 %[n], %[n], 0\n\t"
                         "s_min_u32 s64, s64, s78\n\ts_min_u32 s64, s64, s79\n\ts_cmp_ge_u32 s64, %[c]\n\ts_addc_u32 %[n], %[n], 0\n\t"
                         "s_min_u32 %[n], %[n], 128\n"
                         "LPB%=:\n\ts_waitcnt lgkmcnt(0)\n\ts_cmp_ge_u32 %[rb], %[wl]\n\ts_cbranch_scc1 LPC%=\n\t"
                         "s_cmp_ge_u32 s80, %[c]\n\ts_addc_u32 %[m], %[m], 0\n\t"
                         "s_min_u32 s80, s80, s81\n\ts_cmp_ge_u32 s80, %[c]\n\ts_addc_u32 %[m], %[m], 0\n\t"
                         "s_min_u32 s80, s80, s82\n\ts_cmp_ge_u32 s80, %[c]\n\ts_addc_u32 %[m], %[m], 0\n\t"
                         "s_min_u32 s80, s80, s83\n\ts_cmp_ge_u32 s80, %[c]\n\ts_addc_u32 %[m], %[m], 0\n\t"
                         "s_min_u32 s80, s80, s84\n\ts_cmp_ge_u32 s80, %[c]\n\ts_addc_u32 %[m], %[m], 0\n\t"
                         "s_min_u32 s80, s80, s85\n\ts_cmp_ge_u32 s80, %[c]\n\ts_addc_u32 %[m], %[m], 0\n\t"
                         "s_min_u32 s80, s80, s86\n\ts_cmp_ge_u32 s80, %[c]\n\ts_addc_u32 %[m], %[m], 0\n\t"
                         "s_min_u32 s80, s80, s87\n\ts_cmp_ge_u32 s80, %[c]\n\ts_addc_u32 %[m], %[m], 0\n\t"
                         "s_min_u32 s80, s80, s88\n\ts_cmp_ge_u32 s80, %[c]\n\ts_addc_u32 %[m], %[m], 0\n\t"
                         "s_min_u32 s80, s80, s89\n\ts_cmp_ge_u32 s80, %[c]\n\ts_addc_u32 %[m], %[m], 0\n\t"
                         "s_min_u32 s80, s80, s90\n\ts_cmp_ge_u32 s80, %[c]\n\ts_addc_u32 %[m], %[m], 0\n\t"
                         "s_min_u32 s80, s80, s91\n\ts_cmp_ge_u32 s80, %[c]\n\ts_addc_u32 %[m], %[m], 0\n\t"
                         "s_min_u32 s80, s80, s92\n\ts_cmp_ge_u32 s80, %[c]\n\ts_addc_u32 %[m], %[m], 0\n\t"
                         "s_min_u32 s80, s80, s93\n\ts_cmp_ge_u32 s80, %[c]\n\ts_addc_u32 %[m], %[m], 0\n\t"
                         "s_min_u32 s80, s80, s94\n\ts_cmp_ge_u32 s80, %[c]\n\ts_addc_u32 %[m], %[m], 0\n\t"
                         "s_min_u32 s80, s80, s95\n\ts_cmp_ge_u32 s80, %[c]\n\ts_addc_u32 %[m], %[m], 0\n\t"
                         "\nLPC%=:"
                         : [n] "+&s"(na), [m] "+&s"(nb) : [p] "s"(fx + na * 2), [q] "s"(fw + (rb & ~15u)), [c] "s"(call), [rb] "s"(rb), [wl] "s"(wl) : "memory", "scc", POLL_CLOB);
            if (na != ra || nb != rb) {
                ra = na; rb = nb;
                lds_st(apub, (lane & 1) ? rb : (ra >= (unsigned)NST ? BIG : ra + (RING + 1 - XD)));
                if (ra >= 24u && ra < PQ1) __builtin_amdgcn_s_sleep(PQS); else __builtin_amdgcn_s_sleep(PQF);
            } else {
                if (give_up(tmo, call, t0, spins)) { dead = true; break; }
                if (ra >= (unsigned)NST) __builtin_amdgcn_s_sleep(16); else __builtin_amdgcn_s_sleep(2);
            }
        }
        if (dead && lane < 2) lds_st(lbase + (lane ? L_RDYB : L_GO), BIG);
        while (PZ && !dead) {
            unsigned f;
            asm volatile("s_load_dword %0, %1, 0x0 glc\n\ts_waitcnt lgkmcnt(0)" : "=s"(f) : "s"(ws + WS_PZ + (mt * 4 + min(nt >> 2, 2)) * 4) : "memory");
            if (f >= call) break;
            dead = give_up(tmo, call, t0, spins);
            __builtin_amdgcn_s_sleep(16);
        }
        while (lds_ld(lbase + L_WMAX + 4) < 4u) __builtin_amdgcn_s_sleep(1);
        const u64* gp = lane == 16 ? (const u64*)tmo : (const u64*)(ws + WS_AMG + ((call & 1) * 2 * NR + (lane & 15)) * 128);
        t0 = rt();
        unsigned told = ~0u, rounds = 0;
        const unsigned own = lds_ld(lbase + L_WMAX);
        for (unsigned fin = 3u; fin == 3u;) {
            const u64 e = __hip_atomic_load(gp, RLX_SYS);
            const bool ok = lane < 16 && (unsigned)(e >> 32) == call;
            const unsigned in = (unsigned)__ballot(ok);
            if (dead || (unsigned)__builtin_amdgcn_readlane((int)(unsigned)e, 16) == call || give_up(tmo, call, t0, spins)) fin = 2u;
            else if ((in & 0xffu) == 0xffu) fin = 1u;
            if (fin == 3u && __builtin_popcount((in | in >> 8) & 0xffu) < SPECR) continue;
            const unsigned m = wave_max_u32(ok ? (unsigned)e : 0u);
            if (fin == 3u && (m == told || rounds >= ROUNDS || m < own)) continue;
            ++rounds;
            told = m;
            const float amax = __uint_as_float(m);
            const u32x2 fw2 = {__float_as_uint(amax > 0.f ? 127.0f / amax : 0.f), fin};
            if (lane == 0) asm volatile("ds_write_b64 %0, %1" :: "v"(lbase + L_FIN), "v"(fw2) : "memory");
        }
    }
}
}

extern "C" void kernel_launch(int rank, int nranks, ncclComm_t comm,
                              hipStream_t stream,
                              void* const* d_in, const int* in_sizes, int n_in,
                              void* d_out, int out_size,
                              const int* in_dtypes, int out_dtype,
                              void* const* d_out_all,
                              void* const* d_ws_all, size_t ws_bytes) {
    (void)comm; (void)in_sizes; (void)n_in; (void)out_size; (void)in_dtypes; (void)out_dtype; (void)d_out_all;
    if (nranks != agk::NR || (unsigned)rank >= (unsigned)agk::NR || ws_bytes < agk::WS_END) { fprintf(stderr, "unsupported configuration\n"); return; }
    agk::Params p;
    p.x = (const float*)d_in[0]; p.w = (const float*)d_in[1]; p.out = (signed char*)d_out;
    for (int i = 0; i < agk::NR; ++i) p.ws[i] = (char*)d_ws_all[i];
    p.own = p.ws[rank]; p.rank = rank; p.pad = 0;
    agk::ag_gemm_kernel<<<agk::NWG, 512, 0, stream>>>(p);
    HIP_CHECK(hipGetLastError());
}
```
